```python
import numpy as np

kernel_name = "gpu_gemm_a2a_m2048_k2048_n2048_f32_gelu"


ABSMAX_RTOL = 0.001

COMPUTE_DTYPE = "float32"
EPILOGUE = "gelu"
IN_QUANT = None
OUT_QUANT = None
IN_DTYPES = {"x": "float32", "w_mat": "float32"}
WORLD_SIZE = 8

def setup_inputs():

    x = np.random.randn(2048, 2048).astype(np.float32)
    w_mat = (np.random.randn(2048, 2048) / np.sqrt(2048)).astype(np.float32)
    return {"x": x, "w_mat": w_mat}

def _epilogue(y):
    c = 0.7978845608028654
    return (0.5 * y * (1.0 + np.tanh(c * (y + 0.044715 * y ** 3)))).astype(np.float32)

def reference(x, w_mat):
    y = (x @ w_mat).astype(np.float32)
    return _epilogue(y)
```

```cpp
#include <hip/hip_runtime.h>
#include <rccl/rccl.h>
#include <cstdio>
#include <cstdint>

#define HIP_CHECK(expr) do { hipError_t _e = (expr); if (_e != hipSuccess) { \
    fprintf(stderr, "HIP error %s at %s:%d\n", hipGetErrorString(_e), __FILE__, __LINE__); \
    return; } } while (0)

namespace {

typedef float f32x4 __attribute__((ext_vector_type(4)));
typedef float f32x2 __attribute__((ext_vector_type(2)));
typedef _Float16 f16x2 __attribute__((ext_vector_type(2)));
typedef _Float16 f16x8 __attribute__((ext_vector_type(8)));
typedef unsigned u32x4 __attribute__((ext_vector_type(4)));
typedef int i32x4 __attribute__((ext_vector_type(4)));

constexpr int NR = 8;
constexpr int MLOC = 256;
constexpr int KDIM = 2048;
constexpr int NDIM = 2048;
constexpr int NLOC = 256;
constexpr int NWG = 256;
constexpr int NTHREADS = 512;

constexpr size_t WS_CNT = 0;
constexpr size_t WS_FLAG = 32768;
constexpr size_t WS_TMO = 65536;
constexpr size_t WS_END = 65536 + 128;

constexpr unsigned long long DEADLINE_TICKS = 100000000ull;

struct Params {
    const float* x;
    const float* w;
    float* out[NR];
    char* ws[NR];
    int rank;
};

constexpr int RING_BYTES = 8 * 16384;
constexpr int RED_BYTES = 2 * 16384;
constexpr int LDS_BYTES = RING_BYTES + RED_BYTES;

__device__ __forceinline__ unsigned pk_f16(float a, float b) {
    f32x2 v = {a, b};
    f16x2 h = __builtin_convertvector(v, f16x2);
    return __builtin_bit_cast(unsigned, h);
}

__device__ __forceinline__ i32x4 make_rsrc(const void* p, unsigned bytes) {
    unsigned long long a = (unsigned long long)p;
    i32x4 r;
    r[0] = (int)(unsigned)a;
    r[1] = (int)(unsigned)(a >> 32);
    r[2] = (int)bytes;
    r[3] = 0x00020000;
    return r;
}

#define WAIT_VM(n) asm volatile("s_waitcnt vmcnt(%0)" :: "i"(n) : "memory")

__device__ __forceinline__ void dma16(unsigned lds_dst, unsigned voff, i32x4 rsrc, unsigned soff) {
    asm volatile("s_waitcnt lgkmcnt(0)\n\t"
                 "s_mov_b32 m0, %0\n\t"
                 "s_nop 0\n\t"
                 "buffer_load_dwordx4 %1, %2, %3 offen lds"
                 :: "s"(lds_dst), "v"(voff), "s"(rsrc), "s"(soff) : "memory");
}

__device__ __forceinline__ f32x4 bload16(unsigned voff, i32x4 rsrc, unsigned soff) {
    f32x4 v;
    asm volatile("buffer_load_dwordx4 %0, %1, %2, %3 offen"
                 : "=v"(v) : "v"(voff), "s"(rsrc), "s"(soff) : "memory");
    return v;
}

__device__ __forceinline__ float gelu_tanh(float y) {
    const float c = 0.7978845608028654f;
    float u = c * (y + 0.044715f * y * y * y);
    float e = __builtin_amdgcn_exp2f(u * (-2.0f * 1.4426950408889634f));
    return y * __builtin_amdgcn_rcpf(1.0f + e);
}

__global__ void __launch_bounds__(NTHREADS, 2) gemm_a2a_kernel(Params p) {
    __shared__ __attribute__((aligned(16))) char lds[LDS_BYTES];

    const int tid = threadIdx.x;
    const int lane = tid & 63;
    const int wave = __builtin_amdgcn_readfirstlane(tid >> 6);
    const int bid = blockIdx.x;
    const int rank = p.rank;
    const int dst = (bid + rank) & 7;
    const int idx = bid >> 3;
    const int ct = idx & 7;
    const int rt = idx >> 3;
    const int col0 = dst * NLOC + ct * 32;
    const int row0 = rt * 64;

    const i32x4 rw = make_rsrc(p.w, KDIM * NDIM * 4);
    const i32x4 rx = make_rsrc(p.x, MLOC * KDIM * 4);

    const int c8 = lane & 7, b3 = (lane >> 3) & 1, g = lane >> 4;

    const unsigned w_voff = (unsigned)((16 * b3 + 4 * g) * (NDIM * 4) + c8 * 16);
    const unsigned w_soff0 = (unsigned)((wave * 256) * (NDIM * 4) + col0 * 4);
    const int r8 = lane >> 3;
    const unsigned x_voff0 = (unsigned)(r8 * (KDIM * 4) + ((c8 ^ (r8 >> 1)) * 16));
    const unsigned x_voff1 = (unsigned)((8 + r8) * (KDIM * 4) + ((c8 ^ (4 + (r8 >> 1))) * 16));
    const unsigned x_soff0 = (unsigned)(row0 * (KDIM * 4) + wave * 1024);
    const unsigned lds_base = (unsigned)(uintptr_t)(__attribute__((address_space(3))) char*)lds;
    const unsigned ring = lds_base + wave * 16384;

    const int fr = lane & 15;
    const int frag_off0 = wave * 16384 + (fr >> 3) * 1024 + (fr & 7) * 128 + ((g ^ (fr >> 1)) * 16);
    const int frag_off1 = wave * 16384 + (fr >> 3) * 1024 + (fr & 7) * 128 + (((g + 4) ^ (fr >> 1)) * 16);

    f32x4 wf[8][4];
#pragma unroll
    for (int kb = 0; kb < 8; ++kb) {
#pragma unroll
        for (int i = 0; i < 4; ++i)
            wf[kb][i] = bload16(w_voff, rw, w_soff0 + (unsigned)((kb * 32 + i) * (NDIM * 4)));
        dma16(ring + kb * 2048, x_voff0, rx, x_soff0 + kb * 128);
        dma16(ring + kb * 2048 + 1024, x_voff1, rx, x_soff0 + kb * 128);
    }
    unsigned* cntp = (unsigned*)(p.ws[rank] + WS_CNT + (size_t)bid * 128);
    unsigned cnt;
    asm volatile("global_load_dword %0, %1, off sc0 sc1" : "=v"(cnt) : "v"(cntp) : "memory");

    float* outp = p.out[dst];
    const i32x4 ro = make_rsrc(outp, (unsigned)(NR * MLOC * NLOC * 4));

    f16x8 wfrag[8][2];

#pragma unroll
    for (int c = 0; c < 4; ++c) {
        f32x4 acc0 = {0.f, 0.f, 0.f, 0.f}, acc1 = {0.f, 0.f, 0.f, 0.f};
#pragma unroll
        for (int kb = 0; kb < 8; ++kb) {
            if (c == 0) {
                asm volatile("s_waitcnt vmcnt(%4)"
                             : "+v"(wf[kb][0]), "+v"(wf[kb][1]), "+v"(wf[kb][2]), "+v"(wf[kb][3])
                             : "i"(43 - 4 * kb) : "memory");
                unsigned pk[4][2];
#pragma unroll
                for (int j = 0; j < 4; ++j) {
                    pk[j][0] = pk_f16(wf[kb][0][j], wf[kb][1][j]);
                    pk[j][1] = pk_f16(wf[kb][2][j], wf[kb][3][j]);
                }
#pragma unroll
                for (int m = 0; m < 2; ++m) {
                    u32x4 f;
#pragma unroll
                    for (int t = 0; t < 2; ++t) {
                        const unsigned P = pk[m][t], Q = pk[2 + m][t];
                        f[t]     = __builtin_amdgcn_update_dpp(P, Q, 0x128, 0xf, 0xc, false);
                        f[2 + t] = __builtin_amdgcn_update_dpp(Q, P, 0x128, 0xf, 0x3, false);
                    }
                    wfrag[kb][m] = __builtin_bit_cast(f16x8, f);
                }
            } else if (c < 3) {
                WAIT_VM(15);
            } else {
                WAIT_VM(15 - 2 * kb);
            }
            const f32x4 a0 = *(const f32x4*)(lds + frag_off0 + kb * 2048);
            const f32x4 a1 = *(const f32x4*)(lds + frag_off1 + kb * 2048);
            u32x4 af;
            af[0] = pk_f16(a0[0], a0[1]);
            af[1] = pk_f16(a0[2], a0[3]);
            af[2] = pk_f16(a1[0], a1[1]);
            af[3] = pk_f16(a1[2], a1[3]);
            if (c < 3) {
                const unsigned so = x_soff0 + (unsigned)((c + 1) * 16 * (KDIM * 4) + kb * 128);
                dma16(ring + kb * 2048, x_voff0, rx, so);
                dma16(ring + kb * 2048 + 1024, x_voff1, rx, so);
            }
            const f16x8 a = __builtin_bit_cast(f16x8, af);
            acc0 = __builtin_amdgcn_mfma_f32_16x16x32_f16(a, wfrag[kb][0], acc0, 0, 0, 0);
            acc1 = __builtin_amdgcn_mfma_f32_16x16x32_f16(a, wfrag[kb][1], acc1, 0, 0, 0);
        }
        char* red = lds + RING_BYTES + (c & 1) * 16384;
        {
            char* wp = red + wave * 2048 + (4 * g) * 128 + (4 * c8 + 2 * b3) * 4;
#pragma unroll
            for (int r = 0; r < 4; ++r) {
                f32x2 v = {acc0[r], acc1[r]};
                *(f32x2*)(wp + r * 128) = v;
            }
        }
        asm volatile("s_waitcnt lgkmcnt(0)" ::: "memory");
        __builtin_amdgcn_s_barrier();
        asm volatile("" ::: "memory");
        {
            const char* rp = red + wave * 256 + lane * 4;
            float s = 0.f;
#pragma unroll
            for (int q = 0; q < 8; ++q) s += *(const float*)(rp + q * 2048);
            const float y = gelu_tanh(s);
            const unsigned yb = __builtin_bit_cast(unsigned, y);
            u32x4 o;
            o[0] = yb;
            o[1] = __builtin_amdgcn_update_dpp(0u, yb, 0x55, 0xf, 0xf, false);
            o[2] = __builtin_amdgcn_update_dpp(0u, yb, 0xaa, 0xf, 0xf, false);
            o[3] = __builtin_amdgcn_update_dpp(0u, yb, 0xff, 0xf, 0xf, false);
            const int orow = rank * MLOC + row0 + c * 16 + wave * 2 + (lane >> 5);
            const unsigned ooff = (lane & 3) ? 0x80000000u
                                             : (unsigned)((orow * NLOC + ct * 32 + (lane & 31)) * 4);
            asm volatile("buffer_store_dwordx4 %0, %1, %2, 0 offen\n\ts_nop 1"
                         :: "v"(o), "v"(ooff), "s"(ro) : "memory");
        }
    }

    asm volatile("s_waitcnt vmcnt(0)" : "+v"(cnt) :: "memory");
    __builtin_amdgcn_s_barrier();
    asm volatile("" ::: "memory");
    const unsigned call = cnt + 1u;
    if (wave == 0) {
        if (dst != rank) {
            if (lane == 0) {
                __builtin_amdgcn_fence(__ATOMIC_RELEASE, "");
                asm volatile("s_waitcnt vmcnt(0)" ::: "memory");
                unsigned* fl = (unsigned*)(p.ws[dst] + WS_FLAG + (size_t)(rank * 32 + idx) * 128);
                __hip_atomic_store(fl, call, __ATOMIC_RELAXED, __HIP_MEMORY_SCOPE_SYSTEM);
                __hip_atomic_store(cntp, call, __ATOMIC_RELAXED, __HIP_MEMORY_SCOPE_SYSTEM);
            }
        } else {
            if (lane == 0) __hip_atomic_store(cntp, call, __ATOMIC_RELAXED, __HIP_MEMORY_SCOPE_SYSTEM);
            if (lane < NR && lane != rank) {
                unsigned* fl = (unsigned*)(p.ws[rank] + WS_FLAG + (size_t)(lane * 32 + idx) * 128);
                unsigned* tmo = (unsigned*)(p.ws[rank] + WS_TMO);
                const unsigned long long t0 = __builtin_amdgcn_s_memrealtime();
                bool bad = false;
                while ((int)(__hip_atomic_load(fl, __ATOMIC_RELAXED, __HIP_MEMORY_SCOPE_SYSTEM) - call) < 0) {
                    if (__hip_atomic_load(tmo, __ATOMIC_RELAXED, __HIP_MEMORY_SCOPE_AGENT) == call ||
                        __builtin_amdgcn_s_memrealtime() - t0 > DEADLINE_TICKS) {
                        __hip_atomic_store(tmo, call, __ATOMIC_RELAXED, __HIP_MEMORY_SCOPE_AGENT);
                        bad = true;
                        break;
                    }
                    __builtin_amdgcn_s_sleep(1);
                }
                if (bad) {
                    outp[(size_t)(rank * MLOC + row0) * NLOC + ct * 32] = __builtin_nanf("");
                }
            }
        }
    }
}

}

extern "C" void kernel_launch(int rank, int nranks, ncclComm_t comm,
                              hipStream_t stream,
                              void* const* d_in, const int* in_sizes, int n_in,
                              void* d_out, int out_size,
                              const int* in_dtypes, int out_dtype,
                              void* const* d_out_all,
                              void* const* d_ws_all, size_t ws_bytes) {
    if (nranks != NR || n_in < 2 || ws_bytes < WS_END || (unsigned)rank >= (unsigned)NR) {
        fprintf(stderr, "kernel_launch: unexpected configuration\n");
        return;
    }
    Params p;
    p.x = (const float*)d_in[0];
    p.w = (const float*)d_in[1];
    for (int i = 0; i < NR; ++i) {
        p.out[i] = (float*)d_out_all[i];
        p.ws[i] = (char*)d_ws_all[i];
    }
    p.rank = rank;
    gemm_a2a_kernel<<<dim3(NWG), dim3(NTHREADS), 0, stream>>>(p);
    HIP_CHECK(hipGetLastError());
}
```

```cpp
#include <hip/hip_runtime.h>
#include <rccl/rccl.h>
#include <cstdio>
#include <cstdint>

#define HIP_CHECK(expr) do { hipError_t _e = (expr); if (_e != hipSuccess) { \
    fprintf(stderr, "HIP error %s at %s:%d\n", hipGetErrorString(_e), __FILE__, __LINE__); \
    return; } } while (0)

namespace {

typedef float f32x4 __attribute__((ext_vector_type(4)));
typedef float f32x2 __attribute__((ext_vector_type(2)));
typedef _Float16 f16x2 __attribute__((ext_vector_type(2)));
typedef _Float16 f16x8 __attribute__((ext_vector_type(8)));
typedef unsigned u32x4 __attribute__((ext_vector_type(4)));
typedef int i32x4 __attribute__((ext_vector_type(4)));

constexpr int NR = 8;
constexpr int MLOC = 256;
constexpr int KDIM = 2048;
constexpr int NDIM = 2048;
constexpr int NLOC = 256;
constexpr int NWG = 256;
constexpr int NTHREADS = 576;

constexpr size_t WS_CNT = 0;
constexpr size_t WS_FLAG = 32768;
constexpr size_t WS_TMO = 65536;
constexpr size_t WS_END = 65536 + 128;

constexpr unsigned long long DEADLINE_TICKS = 100000000ull;

struct Params {
    const float* x;
    const float* w;
    float* out[NR];
    char* ws[NR];
    int rank;
};

#define K_STAG 0
#define K_LAG 2
#define K_EXP 0
#define K_T1 2
#define K_T2AT 0
constexpr int RPC = 8;
constexpr int NCH = 64 / RPC;
constexpr int XS_BYTES = 8 * 8 * 512;
constexpr int RED_BUF = 8 * RPC * 128;
constexpr int LDS_BYTES = 84 * 1024;
static_assert(XS_BYTES + 2 * RED_BUF <= LDS_BYTES, "lds");

__device__ __forceinline__ unsigned pk_f16(float a, float b) {
    f32x2 v = {a, b};
    f16x2 h = __builtin_convertvector(v, f16x2);
    return __builtin_bit_cast(unsigned, h);
}

__device__ __forceinline__ i32x4 make_rsrc(const void* p, unsigned bytes) {
    unsigned long long a = (unsigned long long)p;
    i32x4 r;
    r[0] = (int)(unsigned)a;
    r[1] = (int)(unsigned)(a >> 32);
    r[2] = (int)bytes;
    r[3] = 0x00020000;
    return r;
}

#define STAMP(i) do {} while (0)
#define CSTAMP(i) do {} while (0)

#define WAIT_VM(n) asm volatile("s_waitcnt vmcnt(%0)" :: "i"(n) : "memory")

#define BLOAD(dst, voff, rsrc, soff) asm volatile("buffer_load_dwordx4 %0, %1, %2, %3 offen" \
        : "=v"(dst) : "v"(voff), "s"(rsrc), "s"(soff))
#define BRELOAD(dst, voff, rsrc, soff) asm volatile("buffer_load_dwordx4 %0, %1, %2, %3 offen" \
        : "+v"(dst) : "v"(voff), "s"(rsrc), "s"(soff))

__device__ __forceinline__ float gelu_tanh(float y) {
    const float c = 0.7978845608028654f;
    float u = c * (y + 0.044715f * y * y * y);
    float e = __builtin_amdgcn_exp2f(u * (-2.0f * 1.4426950408889634f));
    return y * __builtin_amdgcn_rcpf(1.0f + e);
}

__global__ void __launch_bounds__(NTHREADS) gemm_a2a_kernel(Params p) {
    __shared__ __attribute__((aligned(16))) char lds[LDS_BYTES];

    STAMP(0);
    const int tid = threadIdx.x;
    const int lane = tid & 63;
    const int wave = __builtin_amdgcn_readfirstlane(tid >> 6);
    const int bid = blockIdx.x;
    const int rank = p.rank;
    const float* const px = p.x;
    const float* const pw = p.w;
    asm volatile("" :: "s"(px), "s"(pw), "s"(rank));
    const int dst = (bid + rank) & 7;
    const int idx = bid >> 3;
    const int ct = idx & 7;
    const int rt = idx >> 3;
    const int col0 = dst * NLOC + ct * 32;
    const int row0 = rt * 64;

    if (wave < 8) {
        const i32x4 rw = make_rsrc(pw, KDIM * NDIM * 4);
        const i32x4 rx = make_rsrc(px, MLOC * KDIM * 4);
        const int c8 = lane & 7, b3 = (lane >> 3) & 1, g = lane >> 4, r8 = lane >> 3;
        const unsigned w_voff = (unsigned)((8 * g + 4 * b3) * (NDIM * 4) + c8 * 16);
        const unsigned w_soff0 = (unsigned)((wave * 256) * (NDIM * 4) + col0 * 4);
        const unsigned x_voff = (unsigned)(r8 * (KDIM * 4) + c8 * 16);
        const unsigned x_soff0 = (unsigned)(row0 * (KDIM * 4) + wave * 1024);
        char* const xs_w = lds + wave * 4096 + r8 * 64 + (((c8 >> 1) ^ (2 * (r8 >> 2))) * 16) + (c8 & 1) * 8;
        const char* const xs_r = lds + wave * 4096 + c8 * 64 + ((g ^ (2 * (c8 >> 2))) * 16);
        char* const red_w = lds + XS_BYTES + wave * (RPC * 128) + (4 * g) * 128 + (4 * c8 + 2 * b3) * 4;

        const unsigned perm = rt == 0 ? 0x76532410u : rt == 1 ? 0x67423501u : rt == 2 ? 0x75463102u : 0x64572013u;
        const unsigned koff = wave + ct;
#define KB(s) (((perm >> (4 * (s))) + koff) & 7u)
#define LOAD_W(s) do { _Pragma("unroll") for (int i = 0; i < 4; ++i) \
            BLOAD(wf[s][i], w_voff, rw, w_soff0 + (KB(s) * 32 + i) * (unsigned)(NDIM * 4)); } while (0)
#define X_STAGE(s) do { typedef unsigned u32x2 __attribute__((ext_vector_type(2))); \
            u32x2 h_ = {pk_f16(xv[s][0], xv[s][1]), pk_f16(xv[s][2], xv[s][3])}; \
            *(u32x2*)(xs_w + (s) * 512) = h_; } while (0)
#define X_MMA(s) do { const f16x8 a_ = *(const f16x8*)(xs_r + (s) * 512); \
            acc0 = __builtin_amdgcn_mfma_f32_16x16x32_f16(a_, wfrag[s][0], acc0, 0, 0, 0); \
            acc1 = __builtin_amdgcn_mfma_f32_16x16x32_f16(a_, wfrag[s][1], acc1, 0, 0, 0); } while (0)
        const unsigned t_voff1 = r8 < K_T1 ? (unsigned)((8 * r8 + ct) * (KDIM * 4) + c8 * 128) : 0x80000000u;

#define W_CONV(dstf, src) do { unsigned pk[4][2]; \
            _Pragma("unroll") for (int j = 0; j < 4; ++j) { \
                pk[j][0] = pk_f16((src)[0][j], (src)[1][j]); pk[j][1] = pk_f16((src)[2][j], (src)[3][j]); } \
            _Pragma("unroll") for (int m = 0; m < 2; ++m) { u32x4 f; \
                _Pragma("unroll") for (int t = 0; t < 2; ++t) { const unsigned P = pk[m][t], Q = pk[2 + m][t]; \
                    f[t]     = __builtin_amdgcn_update_dpp(P, Q, 0x128, 0xf, 0xc, false); \
                    f[2 + t] = __builtin_amdgcn_update_dpp(Q, P, 0x128, 0xf, 0x3, false); } \
                (dstf)[m] = __builtin_bit_cast(f16x8, f); } } while (0)
#define STAGE_V(v, s) do { typedef unsigned u32x2 __attribute__((ext_vector_type(2))); \
            u32x2 h_ = {pk_f16((v)[0], (v)[1]), pk_f16((v)[2], (v)[3])}; \
            *(u32x2*)(xs_w + (s) * 512) = h_; } while (0)

        f32x4 wf[8][4];
        f32x4 xv[8];
        f32x4 gen[4][4];
        f16x8 wfrag[8][2];
        float touch;
        asm volatile("buffer_load_dword %0, %1, %2, %3 offen" : "=v"(touch) : "v"(t_voff1), "s"(rx), "s"(x_soff0));
        const bool wfirst = rt < 2;
        const unsigned wmask = (unsigned)((rt - 2) >> 31);
#pragma unroll
        for (int j = 0; j < 4; ++j) {
            const unsigned mk = (j < 2) ? wmask : ~wmask;
#define SEL(a, b) ((b) ^ (((a) ^ (b)) & mk))
            i32x4 rs = rw;
            rs[0] = SEL(rw[0], rx[0]); rs[1] = SEL(rw[1], rx[1]); rs[2] = SEL(rw[2], rx[2]);
            const unsigned vo = SEL(w_voff, x_voff);
#pragma unroll
            for (int i = 0; i < 4; ++i) {
                const unsigned so_w = w_soff0 + (KB(j & 1) * 32 + i) * (unsigned)(NDIM * 4);
                const unsigned so_x = x_soff0 + KB((j & 1) * 4 + i) * 128;
                const unsigned so = SEL(so_w, so_x);
                BLOAD(gen[j][i], vo, rs, so);
            }
            __builtin_amdgcn_s_barrier();
        }
        LOAD_W(2);
        __builtin_amdgcn_s_barrier();
        LOAD_W(3);
        STAMP(10);
        const unsigned x_soff1 = x_soff0 + RPC * (KDIM * 4);
        {
            f32x4 acc0 = {0.f, 0.f, 0.f, 0.f}, acc1 = {0.f, 0.f, 0.f, 0.f};
            asm volatile("s_waitcnt vmcnt(8)"
                         : "+v"(gen[0][0]), "+v"(gen[0][1]), "+v"(gen[0][2]), "+v"(gen[0][3]),
                           "+v"(gen[1][0]), "+v"(gen[1][1]), "+v"(gen[1][2]), "+v"(gen[1][3]),
                           "+v"(gen[2][0]), "+v"(gen[2][1]), "+v"(gen[2][2]), "+v"(gen[2][3]),
                           "+v"(gen[3][0]), "+v"(gen[3][1]), "+v"(gen[3][2]), "+v"(gen[3][3]), "+v"(touch));
            STAMP(2);
            if (wfirst) {
#pragma unroll
                for (int s = 0; s < 8; ++s) STAGE_V(gen[2 + (s >> 2)][s & 3], s);
                W_CONV(wfrag[0], gen[0]);
                W_CONV(wfrag[1], gen[1]);
            } else {
#pragma unroll
                for (int s = 0; s < 8; ++s) STAGE_V(gen[s >> 2][s & 3], s);
                W_CONV(wfrag[0], gen[2]);
                W_CONV(wfrag[1], gen[3]);
            }
            X_MMA(0);
            X_MMA(1);
            __builtin_amdgcn_sched_barrier(0);
#pragma unroll
            for (int s = 2; s < 8; ++s) {
                if (s < 6) {
                    __builtin_amdgcn_s_barrier();
                    LOAD_W(s + 2);
                } else {
                    BLOAD(xv[2 * s - 12], x_voff, rx, x_soff1 + KB(2 * s - 12) * 128);
                    BLOAD(xv[2 * s - 11], x_voff, rx, x_soff1 + KB(2 * s - 11) * 128);
                }
                asm volatile("s_waitcnt vmcnt(%4)"
                             : "+v"(wf[s][0]), "+v"(wf[s][1]), "+v"(wf[s][2]), "+v"(wf[s][3])
                             : "i"(s < 6 ? 8 : s == 6 ? 6 : 4));
                if (s == 7) STAMP(3);
                W_CONV(wfrag[s], wf[s]);
                X_MMA(s);
                __builtin_amdgcn_sched_barrier(0);
            }
            if (g < 2) {
#pragma unroll
                for (int r = 0; r < 4; ++r) {
                    f32x2 v = {acc0[r], acc1[r]};
                    *(f32x2*)(red_w + r * 128) = v;
                }
            }
            asm volatile("s_waitcnt lgkmcnt(0)" ::: "memory");
            __builtin_amdgcn_s_barrier();
            STAMP(4);
#pragma unroll
            for (int s = 4; s < 8; ++s) BLOAD(xv[s], x_voff, rx, x_soff1 + KB(s) * 128);
        }
#pragma unroll 1
        for (int c = 1; c < NCH; ++c) {
            CSTAMP(0);
            const unsigned nxt = (c < NCH - 1) ? x_soff0 + (unsigned)((c + 1) * RPC * (KDIM * 4)) : 0x40000000u;
            f32x4 acc0 = {0.f, 0.f, 0.f, 0.f}, acc1 = {0.f, 0.f, 0.f, 0.f};
            f16x8 af[8];
#define MMA(s) do { acc0 = __builtin_amdgcn_mfma_f32_16x16x32_f16(af[s], wfrag[s][0], acc0, 0, 0, 0); \
                    acc1 = __builtin_amdgcn_mfma_f32_16x16x32_f16(af[s], wfrag[s][1], acc1, 0, 0, 0); } while (0)
#pragma unroll
            for (int s = 0; s < 8; ++s) {
                asm volatile("s_waitcnt vmcnt(7)" : "+v"(xv[s]));
                X_STAGE(s);
                af[s] = *(const f16x8*)(xs_r + s * 512);
                BRELOAD(xv[s], x_voff, rx, nxt + KB(s) * 128);
                if (s >= K_LAG) MMA(s - K_LAG);
                __builtin_amdgcn_sched_barrier(0);
                CSTAMP(1 + s);
            }
#pragma unroll
            for (int s = 8 - K_LAG; s < 8; ++s) MMA(s);
            if (g < 2) {
                char* wp = red_w + (c & 1) * RED_BUF;
#pragma unroll
                for (int r = 0; r < 4; ++r) {
                    f32x2 v = {acc0[r], acc1[r]};
                    *(f32x2*)(wp + r * 128) = v;
                }
            }
            asm volatile("s_waitcnt lgkmcnt(0)" ::: "memory");
            CSTAMP(9);
            __builtin_amdgcn_s_barrier();
            CSTAMP(10);
        }
        STAMP(7);
        asm volatile("s_waitcnt vmcnt(0)" : "+v"(xv[0]), "+v"(xv[1]), "+v"(xv[2]), "+v"(xv[3]),
                     "+v"(xv[4]), "+v"(xv[5]), "+v"(xv[6]), "+v"(xv[7]));
    } else {
        unsigned* cntp = (unsigned*)(p.ws[rank] + WS_CNT + (size_t)bid * 128);
        const unsigned call = __hip_atomic_load(cntp, __ATOMIC_RELAXED, __HIP_MEMORY_SCOPE_SYSTEM) + 1u;
        float* outp = p.out[dst];
        const i32x4 ro = make_rsrc(outp, (unsigned)(NR * MLOC * NLOC * 4));
        const unsigned ooff = (unsigned)(((rank * MLOC + row0 + (lane >> 3)) * NLOC + ct * 32 + (lane & 7) * 4) * 4);
        const char* const red_r = lds + XS_BYTES + lane * 16;
        __builtin_amdgcn_s_setprio(3);
        const i32x4 rx = make_rsrc(px, MLOC * KDIM * 4);
        float touch = 0.f;
#pragma unroll
        for (int i = 0; i < 9; ++i) __builtin_amdgcn_s_barrier();
#pragma unroll 1
        for (int c = 0; c < NCH; ++c) {
            __builtin_amdgcn_s_barrier();
            asm volatile("" ::: "memory");
            const char* rp = red_r + (c & 1) * RED_BUF;
#pragma unroll
            for (int h = 0; h < RPC / 8; ++h) {
                f32x4 v = *(const f32x4*)(rp + h * 1024);
#pragma unroll
                for (int q = 1; q < 8; ++q) v += *(const f32x4*)(rp + h * 1024 + q * (RPC * 128));
                f32x4 o;
#pragma unroll
                for (int j = 0; j < 4; ++j) o[j] = gelu_tanh(v[j]);
                asm volatile("buffer_store_dwordx4 %0, %1, %2, %3 offen sc0 sc1\n\ts_nop 1"
                             :: "v"(o), "v"(ooff), "s"(ro), "s"((unsigned)((c * RPC + h * 8) * NLOC * 4)) : "memory");
            }
            if (c == K_T2AT) {
#pragma unroll
                for (int r = K_T1; r < 8; ++r)
                    asm volatile("buffer_load_dword %0, %1, %2, %3 offen" : "+v"(touch)
                                 : "v"(lane * 128), "s"(rx), "s"((unsigned)((row0 + 8 * r + ct) * (KDIM * 4))) : "memory");
            }
        }
        asm volatile("s_waitcnt vmcnt(0)" : "+v"(touch) :: "memory");
        STAMP(8);
        if (dst != rank) {
            if (lane == 0) {
                unsigned* fl = (unsigned*)(p.ws[dst] + WS_FLAG + (size_t)(rank * 32 + idx) * 128);
                __hip_atomic_store(fl, call, __ATOMIC_RELAXED, __HIP_MEMORY_SCOPE_SYSTEM);
                __hip_atomic_store(cntp, call, __ATOMIC_RELAXED, __HIP_MEMORY_SCOPE_SYSTEM);
            }
        } else {
            if (lane == 0) __hip_atomic_store(cntp, call, __ATOMIC_RELAXED, __HIP_MEMORY_SCOPE_SYSTEM);
            if (lane < NR && lane != rank) {
                unsigned* fl = (unsigned*)(p.ws[rank] + WS_FLAG + (size_t)(lane * 32 + idx) * 128);
                unsigned* tmo = (unsigned*)(p.ws[rank] + WS_TMO);
                const unsigned long long t0 = __builtin_amdgcn_s_memrealtime();
                bool bad = false;
                while ((int)(__hip_atomic_load(fl, __ATOMIC_RELAXED, __HIP_MEMORY_SCOPE_SYSTEM) - call) < 0) {
                    if (__hip_atomic_load(tmo, __ATOMIC_RELAXED, __HIP_MEMORY_SCOPE_AGENT) == call ||
                        __builtin_amdgcn_s_memrealtime() - t0 > DEADLINE_TICKS) {
                        __hip_atomic_store(tmo, call, __ATOMIC_RELAXED, __HIP_MEMORY_SCOPE_AGENT);
                        bad = true;
                        break;
                    }
                    __builtin_amdgcn_s_sleep(1);
                }
                if (bad) {
                    outp[(size_t)(rank * MLOC + row0) * NLOC + ct * 32] = __builtin_nanf("");
                }
            }
        }
    }
}

}

extern "C" void kernel_launch(int rank, int nranks, ncclComm_t comm,
                              hipStream_t stream,
                              void* const* d_in, const int* in_sizes, int n_in,
                              void* d_out, int out_size,
                              const int* in_dtypes, int out_dtype,
                              void* const* d_out_all,
                              void* const* d_ws_all, size_t ws_bytes) {
    if (nranks != NR || n_in < 2 || ws_bytes < WS_END || (unsigned)rank >= (unsigned)NR) {
        fprintf(stderr, "kernel_launch: unexpected configuration\n");
        return;
    }
    Params p;
    p.x = (const float*)d_in[0];
    p.w = (const float*)d_in[1];
    for (int i = 0; i < NR; ++i) {
        p.out[i] = (float*)d_out_all[i];
        p.ws[i] = (char*)d_ws_all[i];
    }
    p.rank = rank;
    gemm_a2a_kernel<<<dim3(NWG), dim3(NTHREADS), 0, stream>>>(p);
    HIP_CHECK(hipGetLastError());
}
```

```cpp
#include <hip/hip_runtime.h>
#include <rccl/rccl.h>
#include <cstdio>
#include <cstdint>

#define HIP_CHECK(expr) do { hipError_t _e = (expr); if (_e != hipSuccess) { \
    fprintf(stderr, "HIP error %s at %s:%d\n", hipGetErrorString(_e), __FILE__, __LINE__); \
    return; } } while (0)

namespace {

typedef float f32x4 __attribute__((ext_vector_type(4)));
typedef float f32x2 __attribute__((ext_vector_type(2)));
typedef _Float16 f16x2 __attribute__((ext_vector_type(2)));
typedef _Float16 f16x8 __attribute__((ext_vector_type(8)));
typedef unsigned u32x4 __attribute__((ext_vector_type(4)));
typedef unsigned u32x2 __attribute__((ext_vector_type(2)));
typedef int i32x4 __attribute__((ext_vector_type(4)));

constexpr int NR = 8;
constexpr int MLOC = 256;
constexpr int KDIM = 2048;
constexpr int NDIM = 2048;
constexpr int NLOC = 256;
constexpr int NWG = 256;
constexpr int NTHREADS = 576;

constexpr size_t WS_CNT = 0;
constexpr size_t WS_FLAG = 32768;
constexpr size_t WS_TMO = 65536;
constexpr size_t WS_X16 = 1 << 20;
constexpr size_t WS_END = 2 << 20;

constexpr unsigned long long DEADLINE_TICKS = 100000000ull;

struct Params {
    const float* x;
    const float* w;
    char* wsl;
    int rank;
    int pad;
    float* out[NR];
    char* ws[NR];
};

#define K_X16 1
#define K_SCHED 0
#define K_CODE 0
#define K_WARM 0
#define K_HELP 2
#define K_LK 10
#define K_PP 9

constexpr int RPC = 8;
constexpr int NCH = 64 / RPC;
constexpr int NF32 = 2;
constexpr int XS_BYTES = 9 * 4096;
constexpr int RED_OFF = XS_BYTES;
constexpr int RED_BUF = 8 * 2048;
constexpr int MISC_OFF = RED_OFF + 2 * RED_BUF;
constexpr int LDS_BYTES = 84 * 1024;
static_assert(MISC_OFF + 512 <= LDS_BYTES, "lds");

__device__ __forceinline__ unsigned pk_f16(float a, float b) {
    f32x2 v = {a, b};
    f16x2 h = __builtin_convertvector(v, f16x2);
    return __builtin_bit_cast(unsigned, h);
}

__device__ __forceinline__ i32x4 make_rsrc(const void* p, unsigned bytes) {
    unsigned long long a = (unsigned long long)p;
    i32x4 r;
    r[0] = (int)(unsigned)a;
    r[1] = (int)(unsigned)(a >> 32);
    r[2] = (int)bytes;
    r[3] = 0x00020000;
    return r;
}

#define STAMP(i) do {} while (0)

#define BLOAD(dst, voff, rsrc, soff) asm volatile("buffer_load_dwordx4 %0, %1, %2, %3 offen" \
        : "=v"(dst) : "v"(voff), "s"(rsrc), "s"(soff))
#define BRELOAD(dst, voff, rsrc, soff) asm volatile("buffer_load_dwordx4 %0, %1, %2, %3 offen" \
        : "+v"(dst) : "v"(voff), "s"(rsrc), "s"(soff))
#define BSTORE(val, voff, rsrc, soff) asm volatile("buffer_store_dwordx4 %0, %1, %2, %3 offen sc0 sc1\n\ts_nop 1" \
        :: "v"(val), "v"(voff), "s"(rsrc), "s"(soff) : "memory")
#define BARRIER() __builtin_amdgcn_s_barrier()

__device__ __forceinline__ float gelu_tanh(float y) {
    const float c = 0.7978845608028654f;
    float u = c * (y + 0.044715f * y * y * y);
    float e = __builtin_amdgcn_exp2f(u * (-2.0f * 1.4426950408889634f));
    return y * __builtin_amdgcn_rcpf(1.0f + e);
}

__global__ void __launch_bounds__(NTHREADS) gemm_a2a_kernel(Params p) {
    __shared__ __attribute__((aligned(16))) char lds[LDS_BYTES];

    STAMP(0);
    unsigned long long pc0;
    asm volatile("s_getpc_b64 %0" : "=s"(pc0));
    const int tid = threadIdx.x;
    const int lane = tid & 63;
    const int wave = __builtin_amdgcn_readfirstlane(tid >> 6);
    const int bid = blockIdx.x;
    const int rank = p.rank;
    const float* const px = p.x;
    const float* const pw = p.w;
    char* const wsl = p.wsl;
    asm volatile("" :: "s"(px), "s"(pw), "s"(wsl), "s"(rank));
    const int dst = (bid + rank) & 7;
    const int idx = bid >> 3;
    const int ct = idx & 7;
    const int rt = idx >> 3;
    const int col0 = dst * NLOC + ct * 32;
    const int row0 = rt * 64;

    const i32x4 rx = make_rsrc(px, MLOC * KDIM * 4);
    const i32x4 rh = make_rsrc(wsl + WS_X16, MLOC * KDIM * 2);
    const int c8 = lane & 7, b3 = (lane >> 3) & 1, g = lane >> 4, r8 = lane >> 3;
    const unsigned x_voff = (unsigned)(r8 * (KDIM * 4) + c8 * 16);
    char* const xs_w = lds + wave * 4096 + r8 * 64 + (((c8 >> 1) ^ (2 * (r8 >> 2))) * 16) + (c8 & 1) * 8;
#define STAGE_V(v, s) do { u32x2 h_ = {pk_f16((v)[0], (v)[1]), pk_f16((v)[2], (v)[3])}; \
        *(u32x2*)(xs_w + (s) * 512) = h_; } while (0)

    if (__builtin_expect(wave < 8, 1)) {
        const i32x4 rw = make_rsrc(pw, KDIM * NDIM * 4);
        const unsigned w_voff = (unsigned)((8 * g + 4 * b3) * (NDIM * 4) + c8 * 16);
        const unsigned w_soff0 = (unsigned)((wave * 256) * (NDIM * 4) + col0 * 4);
        const unsigned x_soff0 = (unsigned)(row0 * (KDIM * 4) + wave * 1024);
        const char* const xs_r = lds + wave * 4096 + c8 * 64 + ((g ^ (2 * (c8 >> 2))) * 16);
        char* const red_w = lds + RED_OFF + wave * 2048 + (4 * g) * 128 + (4 * c8 + 2 * b3) * 4;

        const bool slow = K_SCHED == 1 || (K_SCHED == 2 && (((((unsigned)(uintptr_t)pw) >> 7) + ct) & 7) == 3);
        const unsigned perm0 = rt == 0 ? 0x76532410u : rt == 1 ? 0x67423501u : rt == 2 ? 0x75463102u : 0x64572013u;
        const unsigned perm1 = rt == 0 ? 0x75364210u : rt == 1 ? 0x17506432u : rt == 2 ? 0x31720654u : 0x53142076u;
        const unsigned perm = slow ? perm1 : perm0;
#define K_KOFF 2
        const unsigned koff = K_KOFF == 0 ? 0 : K_KOFF == 1 ? wave : wave + ct;
#define KB(s) (((perm >> (4 * (s))) + koff) & 7u)
#define LOAD_W(s) do { _Pragma("unroll") for (int i = 0; i < 4; ++i) \
            BLOAD(wf[s][i], w_voff, rw, w_soff0 + (KB(s) * 32 + i) * (unsigned)(NDIM * 4)); } while (0)
#define X_MMA(s) do { const f16x8 a_ = *(const f16x8*)(xs_r + (s) * 512); \
            acc0 = __builtin_amdgcn_mfma_f32_16x16x32_f16(a_, wfrag[s][0], acc0, 0, 0, 0); \
            acc1 = __builtin_amdgcn_mfma_f32_16x16x32_f16(a_, wfrag[s][1], acc1, 0, 0, 0); } while (0)
#define W_CONV(dstf, src) do { unsigned pk[4][2]; \
            _Pragma("unroll") for (int j = 0; j < 4; ++j) { \
                pk[j][0] = pk_f16((src)[0][j], (src)[1][j]); pk[j][1] = pk_f16((src)[2][j], (src)[3][j]); } \
            _Pragma("unroll") for (int m = 0; m < 2; ++m) { u32x4 f; \
                _Pragma("unroll") for (int t = 0; t < 2; ++t) { const unsigned P = pk[m][t], Q = pk[2 + m][t]; \
                    f[t]     = __builtin_amdgcn_update_dpp(P, Q, 0x128, 0xf, 0xc, false); \
                    f[2 + t] = __builtin_amdgcn_update_dpp(Q, P, 0x128, 0xf, 0x3, false); } \
                (dstf)[m] = __builtin_bit_cast(f16x8, f); } } while (0)
#define PUBLISH(c) do { char* wp = red_w + ((c) & 1) * RED_BUF; \
            _Pragma("unroll") for (int r = 0; r < 4; ++r) { f32x2 v = {acc0[r], acc1[r]}; \
                *(f32x2*)(wp + r * 128) = v; } \
            asm volatile("s_waitcnt lgkmcnt(0)" ::: "memory"); BARRIER(); asm volatile("" ::: "memory"); } while (0)

#define WINDOW(k) BARRIER()

        f32x4 wf[8][4];
        f32x4 xv[8];
        f32x4 gen[4][4];
        f16x8 wfrag[8][2];
        float touch;
        const unsigned t_voff = r8 < NF32 ? (unsigned)((8 * r8 + ct) * (KDIM * 4) + c8 * 128) : 0x80000000u;
        if (dst == rank) {
            const int u0 = (idx & 7) * 24 + wave * 3;
            f32x4 pv[6];
#pragma unroll
            for (int i = 0; i < 3; ++i) {
                const int grp = rt * 4 + 1 + ((u0 + i) >> 6), kb = (u0 + i) & 63;
                BLOAD(pv[2 * i], x_voff, rx, (unsigned)(grp * 16 * (KDIM * 4) + kb * 128));
                BLOAD(pv[2 * i + 1], x_voff, rx, (unsigned)((grp * 16 + 8) * (KDIM * 4) + kb * 128));
            }
#pragma unroll
            for (int i = 0; i < 3; ++i) {
                asm volatile("s_waitcnt vmcnt(%2)" : "+v"(pv[2 * i]), "+v"(pv[2 * i + 1]) : "i"(4 - i));
                STAGE_V(pv[2 * i], 2 * i);
                STAGE_V(pv[2 * i + 1], 2 * i + 1);
                const f32x4 f = *(const f32x4*)(xs_r + b3 * 512 + i * 1024);
                BSTORE(f, lane * 16, rh, (unsigned)((rt * 4 + 1) * 65536 + (u0 + i) * 1024));
            }
            asm volatile("s_waitcnt vmcnt(0)" ::: "memory");
            BARRIER();
        }
        STAMP(1);
        asm volatile("buffer_load_dword %0, %1, %2, %3 offen" : "=v"(touch) : "v"(t_voff), "s"(rx), "s"(x_soff0));
        float touch2 = 0.f;
        {
            const unsigned cs = (3u - (((unsigned)(uintptr_t)pw) >> 7)) & 7u;
            const unsigned hv = (rt >= 2 && lane < 16 && (K_HELP == 1 || ((((rt - 2) * 8 + ct) >> 1) + 8 - wave - cs & 7) >= 4)) ? (unsigned)((((rt - 2) * 8 + ct) * 16 + lane) * (NDIM * 4)) : 0x80000000u;
            asm volatile("buffer_load_dword %0, %1, %2, %3 offen" : "=v"(touch2)
                         : "v"(hv), "s"(rw), "s"((unsigned)((wave * 256) * (NDIM * 4) + dst * (NLOC * 4)) + cs * 128u));
        }
        const bool slowx = K_SCHED == 3 && (((((unsigned)(uintptr_t)pw) >> 7) + ct) & 7) == 3;
        const bool wfirst = (slow || rt < 2) && !slowx;
        const unsigned wmask = wfirst ? ~0u : 0u;
#pragma unroll
        for (int j = 0; j < 4; ++j) {
            const unsigned mk = (j < 2) ? wmask : ~wmask;
#define SEL(a, b) ((b) ^ (((a) ^ (b)) & mk))
            i32x4 rs = rw;
            rs[0] = SEL(rw[0], rx[0]); rs[1] = SEL(rw[1], rx[1]); rs[2] = SEL(rw[2], rx[2]);
            const unsigned vo = SEL(w_voff, x_voff);
#pragma unroll
            for (int i = 0; i < 4; ++i) {
                const unsigned so_w = w_soff0 + (KB(j & 1) * 32 + i) * (unsigned)(NDIM * 4);
                const unsigned so_x = x_soff0 + KB((j & 1) * 4 + i) * 128;
                const unsigned so = SEL(so_w, so_x);
                BLOAD(gen[j][i], vo, rs, so);
            }
            WINDOW(j);
        }
        LOAD_W(2);
        WINDOW(4);
        LOAD_W(3);
        STAMP(10);
        const unsigned x_soff1 = x_soff0 + RPC * (KDIM * 4);
        {
            f32x4 acc0 = {0.f, 0.f, 0.f, 0.f}, acc1 = {0.f, 0.f, 0.f, 0.f};
            asm volatile("s_waitcnt vmcnt(8)"
                         : "+v"(gen[0][0]), "+v"(gen[0][1]), "+v"(gen[0][2]), "+v"(gen[0][3]),
                           "+v"(gen[1][0]), "+v"(gen[1][1]), "+v"(gen[1][2]), "+v"(gen[1][3]),
                           "+v"(gen[2][0]), "+v"(gen[2][1]), "+v"(gen[2][2]), "+v"(gen[2][3]),
                           "+v"(gen[3][0]), "+v"(gen[3][1]), "+v"(gen[3][2]), "+v"(gen[3][3]), "+v"(touch), "+v"(touch2));
            STAMP(2);
            if (wfirst) {
#pragma unroll
                for (int s = 0; s < 8; ++s) STAGE_V(gen[2 + (s >> 2)][s & 3], s);
                W_CONV(wfrag[0], gen[0]);
                W_CONV(wfrag[1], gen[1]);
            } else {
#pragma unroll
                for (int s = 0; s < 8; ++s) STAGE_V(gen[s >> 2][s & 3], s);
                W_CONV(wfrag[0], gen[2]);
                W_CONV(wfrag[1], gen[3]);
            }
            X_MMA(0);
            X_MMA(1);
            __builtin_amdgcn_sched_barrier(0);
#pragma unroll
            for (int s = 2; s < 8; ++s) {
                if (s < 6) {
                    WINDOW(s + 3);
                    LOAD_W(s + 2);
                } else {
#pragma unroll
                    for (int j = 0; j < 4; ++j) BLOAD(xv[4 * s - 24 + j], x_voff, rx, x_soff1 + KB(4 * s - 24 + j) * 128);
                    if (s == 7) BARRIER();
                }
                asm volatile("s_waitcnt vmcnt(8)"
                             : "+v"(wf[s][0]), "+v"(wf[s][1]), "+v"(wf[s][2]), "+v"(wf[s][3]));
                if (s == 7) STAMP(3);
                W_CONV(wfrag[s], wf[s]);
                X_MMA(s);
                __builtin_amdgcn_sched_barrier(0);
            }
            PUBLISH(0);
            STAMP(4);
        }
#define MMA(s) do { acc0 = __builtin_amdgcn_mfma_f32_16x16x32_f16(af[s], wfrag[s][0], acc0, 0, 0, 0); \
                    acc1 = __builtin_amdgcn_mfma_f32_16x16x32_f16(af[s], wfrag[s][1], acc1, 0, 0, 0); } while (0)
        const unsigned h_soff0 = (unsigned)((rt * 4) * 65536 + wave * 8192);
        const unsigned h_voff = (unsigned)(lane * 16);
        const char* const xs_r16 = xs_r + b3 * 512;
        f32x4 xh[8];
        unsigned pre;
#define PREFETCH16(j) do { pre = __builtin_amdgcn_readfirstlane(__hip_atomic_load((unsigned*)(lds + MISC_OFF), \
                    __ATOMIC_RELAXED, __HIP_MEMORY_SCOPE_WORKGROUP)); \
            const unsigned hb = pre ? h_soff0 + (j) * 65536u : 0x40000000u; \
            _Pragma("unroll") for (int s = 0; s < 8; ++s) BLOAD(xh[s], h_voff, rh, hb + KB(s) * 1024); } while (0)
#define STEP16(j) do { \
            f32x4 acc0 = {0.f, 0.f, 0.f, 0.f}, acc1 = acc0; \
            if (pre) { \
                _Pragma("unroll") for (int s = 0; s < 8; ++s) { \
                    asm volatile("s_waitcnt vmcnt(%1)" : "+v"(xh[s]) : "i"(7 - s)); \
                    const f16x8 a_ = __builtin_bit_cast(f16x8, xh[s]); \
                    acc0 = __builtin_amdgcn_mfma_f32_16x16x32_f16(a_, wfrag[s][0], acc0, 0, 0, 0); \
                    acc1 = __builtin_amdgcn_mfma_f32_16x16x32_f16(a_, wfrag[s][1], acc1, 0, 0, 0); \
                    __builtin_amdgcn_sched_barrier(0); \
                } \
            } else { \
                asm volatile("s_waitcnt vmcnt(0)" ::: "memory"); \
                __builtin_amdgcn_sched_barrier(0); \
                _Pragma("unroll") for (int hh = 0; hh < 2; ++hh) { \
                    const unsigned xb = x_soff0 + (j) * 16 * (KDIM * 4); \
                    _Pragma("unroll") for (int s = 0; s < 4; ++s) { \
                        BLOAD(xv[2 * s], x_voff, rx, xb + KB(4 * hh + s) * 128); \
                        BLOAD(xv[2 * s + 1], x_voff, rx, xb + 8 * (KDIM * 4) + KB(4 * hh + s) * 128); } \
                    asm volatile("s_waitcnt vmcnt(0)" : "+v"(xv[0]), "+v"(xv[1]), "+v"(xv[2]), "+v"(xv[3]), \
                                 "+v"(xv[4]), "+v"(xv[5]), "+v"(xv[6]), "+v"(xv[7])); \
                    _Pragma("unroll") for (int s = 0; s < 8; ++s) STAGE_V(xv[s], s); \
                    _Pragma("unroll") for (int s = 0; s < 4; ++s) { \
                        const f16x8 a_ = *(const f16x8*)(xs_r16 + s * 1024); \
                        acc0 = __builtin_amdgcn_mfma_f32_16x16x32_f16(a_, wfrag[4 * hh + s][0], acc0, 0, 0, 0); \
                        acc1 = __builtin_amdgcn_mfma_f32_16x16x32_f16(a_, wfrag[4 * hh + s][1], acc1, 0, 0, 0); } \
                    __builtin_amdgcn_sched_barrier(0); \
                } \
            } \
            __builtin_amdgcn_sched_barrier(0); \
            if ((j) < 3) PREFETCH16((j) + 1); \
            PUBLISH((j) + 1); } while (0)
        {
            f32x4 acc0 = {0.f, 0.f, 0.f, 0.f}, acc1 = {0.f, 0.f, 0.f, 0.f};
            f16x8 af[8];
#pragma unroll
            for (int s = 0; s < 8; ++s) {
                asm volatile("s_waitcnt vmcnt(%1)" : "+v"(xv[s]) : "i"(7 - s));
                STAGE_V(xv[s], s);
                af[s] = *(const f16x8*)(xs_r + s * 512);
                if (s >= 2) MMA(s - 2);
                __builtin_amdgcn_sched_barrier(0);
            }
            MMA(6); MMA(7);
            __builtin_amdgcn_sched_barrier(0);
            PREFETCH16(1);
            PUBLISH(1);
        }
        STEP16(1);
        STEP16(2);
        STEP16(3);
        STAMP(7);
    } else {
        __builtin_amdgcn_s_setprio(3);
        const unsigned sink = (unsigned)(uintptr_t)(__attribute__((address_space(3))) char*)lds + MISC_OFF + 256;
        unsigned* cntp = (unsigned*)(wsl + WS_CNT + (size_t)bid * 128);
        unsigned cnt;
        asm volatile("global_load_dword %0, %1, off sc0 sc1" : "=v"(cnt) : "v"(cntp));
        float* outp = p.out[dst];
        char* wsd = p.ws[dst];
        const i32x4 ro = make_rsrc(outp, (unsigned)(NR * MLOC * NLOC * 4));
        const unsigned ooff = (unsigned)(((rank * MLOC + row0 + r8) * NLOC + ct * 32 + c8 * 4) * 4);
        if (lane == 0) __hip_atomic_store((unsigned*)(lds + MISC_OFF), 0u, __ATOMIC_RELAXED, __HIP_MEMORY_SCOPE_WORKGROUP);
        asm volatile("s_waitcnt lgkmcnt(0)" ::: "memory");
        if (dst == rank) {
            BARRIER();
            asm volatile("s_waitcnt vmcnt(0)" : "+v"(cnt));
            if (lane == 0) __hip_atomic_store(cntp + 1, cnt + 1u, __ATOMIC_RELAXED, __HIP_MEMORY_SCOPE_AGENT);
            STAMP(2);
        }
        const unsigned* fp = (const unsigned*)(wsl + WS_CNT + (size_t)(rt * 64 + (lane & 7) * 8) * 128) + 1;
        unsigned ready = 0;
        unsigned orow = 0;
#define LOOK(seen, tag) do { if (K_X16 && __all((int)((seen) - call) >= 0)) { \
            ready = 1; \
              \
            _Pragma("unroll") for (int i = 0; i < 3; ++i) \
                asm volatile("s_mov_b32 m0, %0\n\ts_nop 0\n\tbuffer_load_dword %1, %2, %3 offen lds" \
                             :: "s"(sink), "v"(lane * 128), "s"(rh), \
                                "s"((unsigned)((rt * 4 + 1 + i) * 65536 + ct * 8192)) : "memory"); \
            if (lane == 0) __hip_atomic_store((unsigned*)(lds + MISC_OFF), 1u, __ATOMIC_RELAXED, __HIP_MEMORY_SCOPE_WORKGROUP); \
            LOOK_STAMP(tag); } } while (0)
#define LOOK_STAMP(tag) do {} while (0)
#pragma unroll
        for (int i = 0; i < K_PP; ++i) BARRIER();
        unsigned seen0;
        asm volatile("global_load_dword %0, %1, off sc1" : "=v"(seen0) : "v"(fp));
#pragma unroll
        for (int i = K_PP; i < K_LK; ++i) BARRIER();
        asm volatile("s_waitcnt vmcnt(0)" : "+v"(seen0), "+v"(cnt));
        const unsigned call = __builtin_amdgcn_readfirstlane(cnt) + 1u;
        LOOK(seen0, 2);
#pragma unroll
        for (int i = K_LK; i < 10; ++i) BARRIER();
        STAMP(3);
#pragma unroll 1
        for (int c = 0; c < 5; ++c) {
            BARRIER();
            asm volatile("" ::: "memory");
            const int nh = c >= 2 ? 2 : 1;
            for (int h = 0; h < nh; ++h) {
                const char* rp = lds + RED_OFF + lane * 16 + (c & 1) * RED_BUF + h * 1024;
                f32x4 v = *(const f32x4*)(rp);
#pragma unroll
                for (int q = 1; q < 8; ++q) v += *(const f32x4*)(rp + q * 2048);
                f32x4 o;
#pragma unroll
                for (int j = 0; j < 4; ++j) o[j] = gelu_tanh(v[j]);
                BSTORE(o, ooff, ro, orow);
                orow += RPC * NLOC * 4;
            }
            if (!ready && c < 3) {
                const unsigned seen = __hip_atomic_load(fp, __ATOMIC_RELAXED, __HIP_MEMORY_SCOPE_AGENT);
                LOOK(seen, c + 3);
            }
        }
        asm volatile("s_waitcnt vmcnt(0)" ::: "memory");
        STAMP(8);
        if (dst != rank) {
            if (lane == 0) {
                unsigned* fl = (unsigned*)(wsd + WS_FLAG + (size_t)(rank * 32 + idx) * 128);
                __hip_atomic_store(fl, call, __ATOMIC_RELAXED, __HIP_MEMORY_SCOPE_SYSTEM);
                __hip_atomic_store(cntp, call, __ATOMIC_RELAXED, __HIP_MEMORY_SCOPE_SYSTEM);
            }
        } else {
            if (lane == 0) __hip_atomic_store(cntp, call, __ATOMIC_RELAXED, __HIP_MEMORY_SCOPE_SYSTEM);
            if (lane < NR && lane != rank) {
                unsigned* fl = (unsigned*)(wsl + WS_FLAG + (size_t)(lane * 32 + idx) * 128);
                unsigned* tmo = (unsigned*)(wsl + WS_TMO);
                const unsigned long long t0 = __builtin_amdgcn_s_memrealtime();
                bool bad = false;
                unsigned spins = 0;
                while ((int)(__hip_atomic_load(fl, __ATOMIC_RELAXED, __HIP_MEMORY_SCOPE_SYSTEM) - call) < 0) {
                    if ((++spins & 63u) == 0u) {
                        if (__hip_atomic_load(tmo, __ATOMIC_RELAXED, __HIP_MEMORY_SCOPE_AGENT) == call ||
                            __builtin_amdgcn_s_memrealtime() - t0 > DEADLINE_TICKS) {
                            __hip_atomic_store(tmo, call, __ATOMIC_RELAXED, __HIP_MEMORY_SCOPE_AGENT);
                            bad = true;
                            break;
                        }
                    }
                }
                if (bad) {
                    outp[(size_t)(rank * MLOC + row0) * NLOC + ct * 32] = __builtin_nanf("");
                }
            }
        }
    }
}

}

extern "C" void kernel_launch(int rank, int nranks, ncclComm_t comm,
                              hipStream_t stream,
                              void* const* d_in, const int* in_sizes, int n_in,
                              void* d_out, int out_size,
                              const int* in_dtypes, int out_dtype,
                              void* const* d_out_all,
                              void* const* d_ws_all, size_t ws_bytes) {
    if (nranks != NR || n_in < 2 || ws_bytes < WS_END || (unsigned)rank >= (unsigned)NR) {
        fprintf(stderr, "kernel_launch: unexpected configuration\n");
        return;
    }
    Params p;
    p.x = (const float*)d_in[0];
    p.w = (const float*)d_in[1];
    for (int i = 0; i < NR; ++i) {
        p.out[i] = (float*)d_out_all[i];
        p.ws[i] = (char*)d_ws_all[i];
    }
    p.wsl = p.ws[rank];
    p.rank = rank;
    p.pad = 0;
    gemm_a2a_kernel<<<dim3(NWG), dim3(NTHREADS), 0, stream>>>(p);
    HIP_CHECK(hipGetLastError());
}
```

```cpp
#include <hip/hip_runtime.h>
#include <rccl/rccl.h>
#include <cstdio>
#include <cstdint>

#define HIP_CHECK(expr) do { hipError_t _e = (expr); if (_e != hipSuccess) { \
    fprintf(stderr, "HIP error %s at %s:%d\n", hipGetErrorString(_e), __FILE__, __LINE__); \
    return; } } while (0)

namespace {

typedef float f32x4 __attribute__((ext_vector_type(4)));
typedef float f32x2 __attribute__((ext_vector_type(2)));
typedef _Float16 f16x2 __attribute__((ext_vector_type(2)));
typedef _Float16 f16x8 __attribute__((ext_vector_type(8)));
typedef unsigned u32x4 __attribute__((ext_vector_type(4)));
typedef unsigned u32x2 __attribute__((ext_vector_type(2)));
typedef int i32x4 __attribute__((ext_vector_type(4)));

constexpr int NR = 8;
constexpr int MLOC = 256;
constexpr int KDIM = 2048;
constexpr int NDIM = 2048;
constexpr int NLOC = 256;
constexpr int NWG = 256;
constexpr int NTHREADS = 640;

constexpr size_t WS_CNT = 0;
constexpr size_t WS_FLAG = 32768;
constexpr size_t WS_TMO = 65536;
constexpr size_t WS_X16 = 1 << 20;
constexpr size_t WS_END = 2 << 20;

constexpr unsigned long long DEADLINE_TICKS = 100000000ull;

struct Params {
    const float* x;
    const float* w;
    char* wsl;
    int rank;
    int pad;
    float* out[NR];
    char* ws[NR];
};

#define K_X16 1
#define K_HELP 0
#define K_IMG 1
#define K_TRIES 4
#define K_WARM 1
#define K_LATE 0
#define K_PERM 0

constexpr int XS_BYTES = 8 * 4096;
constexpr int RED_OFF = XS_BYTES;
constexpr int RED_BUF = 8 * 2048;
constexpr int MISC_OFF = RED_OFF + 2 * RED_BUF;
constexpr int LDS_BYTES = 84 * 1024;
static_assert(MISC_OFF + 64 <= LDS_BYTES, "lds");

__device__ __forceinline__ unsigned pk_f16(float a, float b) {
    f32x2 v = {a, b};
    f16x2 h = __builtin_convertvector(v, f16x2);
    return __builtin_bit_cast(unsigned, h);
}

__device__ __forceinline__ i32x4 make_rsrc(const void* p, unsigned bytes) {
    unsigned long long a = (unsigned long long)p;
    i32x4 r;
    r[0] = (int)(unsigned)a;
    r[1] = (int)(unsigned)(a >> 32);
    r[2] = (int)bytes;
    r[3] = 0x00020000;
    return r;
}

#define STAMP(i) do {} while (0)

#define BLOAD(dst, voff, rsrc, soff) asm volatile("buffer_load_dwordx4 %0, %1, %2, %3 offen" \
        : "=v"(dst) : "v"(voff), "s"(rsrc), "s"(soff))
#define BRELOAD(dst, voff, rsrc, soff) asm volatile("buffer_load_dwordx4 %0, %1, %2, %3 offen" \
        : "+v"(dst) : "v"(voff), "s"(rsrc), "s"(soff))
#define BSTORE(val, voff, rsrc, soff) asm volatile("buffer_store_dwordx4 %0, %1, %2, %3 offen sc0 sc1\n\ts_nop 1" \
        :: "v"(val), "v"(voff), "s"(rsrc), "s"(soff) : "memory")
#define HSTORE(val, voff, rsrc, soff) asm volatile("buffer_store_dwordx4 %0, %1, %2, %3 offen sc1\n\ts_nop 1" \
        :: "v"(val), "v"(voff), "s"(rsrc), "s"(soff) : "memory")
#define BARRIER() __builtin_amdgcn_s_barrier()
#define TAG_MASK 0x00010001u
#define TAG_OF(call) (((call) & 1u) | (((call) & 2u) << 15))

__device__ __forceinline__ float gelu_tanh(float y) {
    const float c = 0.7978845608028654f;
    float u = c * (y + 0.044715f * y * y * y);
    float e = __builtin_amdgcn_exp2f(u * (-2.0f * 1.4426950408889634f));
    return y * __builtin_amdgcn_rcpf(1.0f + e);
}

__global__ void __launch_bounds__(NTHREADS) gemm_a2a_kernel(Params p) {
    __shared__ __attribute__((aligned(16))) char lds[LDS_BYTES];

    STAMP(0);
    const int tid = threadIdx.x;
    const int lane = tid & 63;
    const int wave = __builtin_amdgcn_readfirstlane(tid >> 6);
    const int bid = blockIdx.x;
    const int rank = p.rank;
    const float* const px = p.x;
    const float* const pw = p.w;
    char* const wsl = p.wsl;
    asm volatile("" :: "s"(px), "s"(pw), "s"(wsl), "s"(rank));
    const int dst = (bid + rank) & 7;
    const int idx = bid >> 3;
    const int ct = idx & 7;
    const int rt = idx >> 3;
    const int col0 = dst * NLOC + ct * 32;
    const int row0 = rt * 64;
    const int c8 = lane & 7, b3 = (lane >> 3) & 1, g = lane >> 4, r8 = lane >> 3;

    if (__builtin_expect(wave < 8, 1)) {
        const i32x4 rw = make_rsrc(pw, KDIM * NDIM * 4);
        const i32x4 rx = make_rsrc(px, MLOC * KDIM * 4);
        const i32x4 rh = make_rsrc(wsl + WS_X16, MLOC * KDIM * 2);
        const unsigned w_voff = (unsigned)((8 * g + 4 * b3) * (NDIM * 4) + c8 * 16);
        const unsigned w_soff0 = (unsigned)((wave * 256) * (NDIM * 4) + col0 * 4);
        const unsigned x_voff = (unsigned)(r8 * (KDIM * 4) + c8 * 16);
        const unsigned x_soff0 = (unsigned)(row0 * (KDIM * 4) + wave * 1024);
        char* const xs_w = lds + wave * 4096 + r8 * 64 + (((c8 >> 1) ^ (2 * (r8 >> 2))) * 16) + (c8 & 1) * 8;
        const char* const xs_r = lds + wave * 4096 + b3 * 512 + c8 * 64 + ((g ^ (2 * (c8 >> 2))) * 16);
        const unsigned rnd_add = (c8 & 1) ? 0u : 0x1fffu, rnd_lsb = (c8 & 1) ? 0u : 1u, rnd_and = (c8 & 1) ? ~0u : ~0x3fffu;
#define RND9(x) __uint_as_float((__float_as_uint(x) + rnd_add + ((__float_as_uint(x) >> 14) & rnd_lsb)) & rnd_and)
#define STAGE_V(v, s) do { const float e0_ = (v)[0], e1_ = (v)[1]; \
            u32x2 h_ = {pk_f16(RND9(e0_), RND9(e1_)), pk_f16((v)[2], (v)[3])}; \
            *(u32x2*)(xs_w + (s) * 512) = h_; } while (0)
        char* const red_w = lds + RED_OFF + wave * 2048 + (4 * g) * 128 + (4 * c8 + 2 * b3) * 4;
        const unsigned h_voff = (unsigned)(lane * 16);
        f32x4 xv[8];

        const unsigned kbp = ct * 8 + wave;
        const unsigned xs = (unsigned)(row0 * (KDIM * 4)) + kbp * 128, hs = (unsigned)(rt * 4 * 65536) + kbp * 1024;
        unsigned cnt;
#define PRODUCE(i) do { STAGE_V(xv[2 * (i)], 2 * (i)); STAGE_V(xv[2 * (i) + 1], 2 * (i) + 1); \
            u32x4 f = *(const u32x4*)(xs_r + (i) * 1024); \
            f[0] = (f[0] & ~TAG_MASK) | TAG_OF(cnt + 1u); \
            HSTORE(f, h_voff, rh, hs + (i) * 65536); } while (0)
        if (dst == rank) {
            asm volatile("global_load_dword %0, %1, off sc0 sc1" : "=v"(cnt) : "v"(wsl + WS_CNT + (size_t)bid * 128));
#pragma unroll
            for (int i = 0; i < 4; ++i) BLOAD(xv[i], x_voff, rx, xs + i * 8 * (KDIM * 4));
            BARRIER();
#pragma unroll
            for (int i = 4; i < 8; ++i) BLOAD(xv[i], x_voff, rx, xs + i * 8 * (KDIM * 4));
            asm volatile("s_waitcnt vmcnt(6)" : "+v"(xv[0]), "+v"(xv[1]), "+v"(cnt));
            PRODUCE(0);
            asm volatile("s_waitcnt vmcnt(5)" : "+v"(xv[2]), "+v"(xv[3]));
            PRODUCE(1);
            STAMP(5);
            asm volatile("s_waitcnt vmcnt(2)" : "+v"(xv[4]), "+v"(xv[5]), "+v"(xv[6]), "+v"(xv[7]));
            PRODUCE(2);
            PRODUCE(3);
        }

        const unsigned perm = rt == 0 ? 0x76532410u : rt == 1 ? 0x67423501u : rt == 2 ? 0x75463102u : 0x64572013u;
        const unsigned koff = wave + ct;
#define KB(s) (((perm >> (4 * (s))) + koff) & 7u)
#define LOAD_W(s) do { _Pragma("unroll") for (int i = 0; i < 4; ++i) \
            BLOAD(wf[s][i], w_voff, rw, w_soff0 + (KB(s) * 32 + i) * (unsigned)(NDIM * 4)); } while (0)
#define W_CONV(dstf, src) do { unsigned pk[4][2]; \
            _Pragma("unroll") for (int j = 0; j < 4; ++j) { \
                pk[j][0] = pk_f16((src)[0][j], (src)[1][j]); pk[j][1] = pk_f16((src)[2][j], (src)[3][j]); } \
            _Pragma("unroll") for (int m = 0; m < 2; ++m) { u32x4 f; \
                _Pragma("unroll") for (int t = 0; t < 2; ++t) { const unsigned P = pk[m][t], Q = pk[2 + m][t]; \
                    f[t]     = __builtin_amdgcn_update_dpp(P, Q, 0x128, 0xf, 0xc, false); \
                    f[2 + t] = __builtin_amdgcn_update_dpp(Q, P, 0x128, 0xf, 0x3, false); } \
                (dstf)[m] = __builtin_bit_cast(f16x8, f); } } while (0)

        f32x4 wf[8][4];
        f16x8 wfrag[8][2];
        u32x4 xh[8];
        const unsigned h_soff0 = (unsigned)((rt * 4) * 65536 + wave * 8192);
        float touch = 0.f;
        STAMP(1);
        LOAD_W(0);
        BARRIER();
        LOAD_W(1);
#pragma unroll
        for (int s = 0; s < 8; ++s) {
            if (s < 6) {
                BARRIER();
                LOAD_W(s + 2);
            } else {
#pragma unroll
                for (int j = 0; j < 4; ++j) BLOAD(xh[4 * s - 24 + j], h_voff, rh, h_soff0 + KB(4 * s - 24 + j) * 1024);
            }
            asm volatile("s_waitcnt vmcnt(8)"
                         : "+v"(wf[s][0]), "+v"(wf[s][1]), "+v"(wf[s][2]), "+v"(wf[s][3]), "+v"(touch));
            if (s == 0) STAMP(2);
            if (s == 7) STAMP(3);
            W_CONV(wfrag[s], wf[s]);
            __builtin_amdgcn_sched_barrier(0);
        }
        const unsigned par = __builtin_amdgcn_readfirstlane(__hip_atomic_load((unsigned*)(lds + MISC_OFF),
                                 __ATOMIC_RELAXED, __HIP_MEMORY_SCOPE_WORKGROUP));
#define ALL_XH "+v"(xh[0]), "+v"(xh[1]), "+v"(xh[2]), "+v"(xh[3]), "+v"(xh[4]), "+v"(xh[5]), "+v"(xh[6]), "+v"(xh[7])
#define STALE() (!K_X16 || __any((((xh[0][0] ^ par) | (xh[1][0] ^ par) | (xh[2][0] ^ par) | (xh[3][0] ^ par) | \
                                  (xh[4][0] ^ par) | (xh[5][0] ^ par) | (xh[6][0] ^ par) | (xh[7][0] ^ par)) & TAG_MASK) != 0))
#define STEP16(j) do { \
            f32x4 acc0 = {0.f, 0.f, 0.f, 0.f}, acc1 = acc0; \
            asm volatile("s_waitcnt vmcnt(0)" : ALL_XH); \
            bool stale = STALE(); \
            for (int tries = 0; stale && tries < K_TRIES; ++tries) {   \
                _Pragma("unroll") for (int s = 0; s < 8; ++s) \
                    asm volatile("buffer_load_dwordx4 %0, %1, %2, %3 offen sc1" : "+v"(xh[s]) \
                                 : "v"(h_voff), "s"(rh), "s"(h_soff0 + (j) * 65536u + KB(s) * 1024)); \
                asm volatile("s_waitcnt vmcnt(0)" : ALL_XH); \
                stale = STALE(); \
                RETRY_STAMP(); \
            } \
            if (!stale) { \
                _Pragma("unroll") for (int s = 0; s < 8; ++s) { \
                    xh[s][0] &= ~TAG_MASK; \
                    const f16x8 a_ = __builtin_bit_cast(f16x8, xh[s]); \
                    acc0 = __builtin_amdgcn_mfma_f32_16x16x32_f16(a_, wfrag[s][0], acc0, 0, 0, 0); \
                    acc1 = __builtin_amdgcn_mfma_f32_16x16x32_f16(a_, wfrag[s][1], acc1, 0, 0, 0); \
                } \
            } else {   \
                _Pragma("unroll") for (int hh = 0; hh < 2; ++hh) { \
                    const unsigned xb = x_soff0 + (j) * 16 * (KDIM * 4); \
                    _Pragma("unroll") for (int s = 0; s < 4; ++s) { \
                        BLOAD(xv[2 * s], x_voff, rx, xb + KB(4 * hh + s) * 128); \
                        BLOAD(xv[2 * s + 1], x_voff, rx, xb + 8 * (KDIM * 4) + KB(4 * hh + s) * 128); } \
                    asm volatile("s_waitcnt vmcnt(0)" : "+v"(xv[0]), "+v"(xv[1]), "+v"(xv[2]), "+v"(xv[3]), \
                                 "+v"(xv[4]), "+v"(xv[5]), "+v"(xv[6]), "+v"(xv[7])); \
                    _Pragma("unroll") for (int s = 0; s < 8; ++s) STAGE_V(xv[s], s); \
                    _Pragma("unroll") for (int s = 0; s < 4; ++s) { \
                        u32x4 f_ = *(const u32x4*)(xs_r + s * 1024); \
                        f_[0] &= ~TAG_MASK;       \
                        const f16x8 a_ = __builtin_bit_cast(f16x8, f_); \
                        acc0 = __builtin_amdgcn_mfma_f32_16x16x32_f16(a_, wfrag[4 * hh + s][0], acc0, 0, 0, 0); \
                        acc1 = __builtin_amdgcn_mfma_f32_16x16x32_f16(a_, wfrag[4 * hh + s][1], acc1, 0, 0, 0); } \
                    __builtin_amdgcn_sched_barrier(0); \
                } \
            } \
            __builtin_amdgcn_sched_barrier(0); \
            if ((j) < 3) {   \
                _Pragma("unroll") for (int s = 0; s < 8; ++s) \
                    BLOAD(xh[s], h_voff, rh, h_soff0 + ((j) + 1) * 65536u + KB(s) * 1024); } \
              \
            char* wp = red_w + ((j) & 1) * RED_BUF; \
            _Pragma("unroll") for (int r = 0; r < 4; ++r) { f32x2 v = {acc0[r], acc1[r]}; \
                *(f32x2*)(wp + r * 128) = v; } \
            asm volatile("s_waitcnt lgkmcnt(0)" ::: "memory"); BARRIER(); asm volatile("" ::: "memory"); } while (0)
#define RETRY_STAMP() do {} while (0)
        STEP16(0);
        STAMP(4);
        STEP16(1);
        STEP16(2);
        STEP16(3);
        STAMP(7);
    } else {
        __builtin_amdgcn_s_setprio(3);
        const int eh = wave - 8;
        unsigned* cntp = (unsigned*)(wsl + WS_CNT + (size_t)bid * 128);
        unsigned cnt;
        asm volatile("global_load_dword %0, %1, off sc0 sc1" : "=v"(cnt) : "v"(cntp));
        float* outp = p.out[dst];
        char* wsd = p.ws[dst];
        const i32x4 ro = make_rsrc(outp, (unsigned)(NR * MLOC * NLOC * 4));
        const unsigned ooff = (unsigned)(((rank * MLOC + row0 + eh * 8 + r8) * NLOC + ct * 32 + c8 * 4) * 4);
        if (dst == rank) BARRIER();
#pragma unroll
        for (int i = 0; i < 4; ++i) BARRIER();
        asm volatile("s_waitcnt vmcnt(0)" : "+v"(cnt));
        const unsigned call = __builtin_amdgcn_readfirstlane(cnt) + 1u;
        if (lane == 0) __hip_atomic_store((unsigned*)(lds + MISC_OFF), TAG_OF(call), __ATOMIC_RELAXED, __HIP_MEMORY_SCOPE_WORKGROUP);
        asm volatile("s_waitcnt lgkmcnt(0)" ::: "memory");
        BARRIER();
        BARRIER();
        STAMP(3);
#pragma unroll 1
        for (int c = -K_WARM; c < 4; ++c) {
            if (!K_WARM && c == 0) BARRIER();
            BARRIER();
            asm volatile("" ::: "memory");
            const char* rp = lds + RED_OFF + lane * 16 + (c & 1) * RED_BUF + eh * 1024;
            f32x4 v = *(const f32x4*)(rp);
#pragma unroll
            for (int q = 1; q < 8; ++q) v += *(const f32x4*)(rp + q * 2048);
            f32x4 o;
#pragma unroll
            for (int j = 0; j < 4; ++j) o[j] = gelu_tanh(v[j]);
            BSTORE(o, ooff, ro, c < 0 ? 0x40000000u : (unsigned)(c * 16 * NLOC * 4));
        }
        if (dst != rank) {
            asm volatile("s_waitcnt vmcnt(0)" ::: "memory");
            BARRIER();
            STAMP(8);
            if (eh == 0 && lane == 0) {
                unsigned* fl = (unsigned*)(wsd + WS_FLAG + (size_t)(rank * 32 + idx) * 128);
                __hip_atomic_store(fl, call, __ATOMIC_RELAXED, __HIP_MEMORY_SCOPE_SYSTEM);
                __hip_atomic_store(cntp, call, __ATOMIC_RELAXED, __HIP_MEMORY_SCOPE_SYSTEM);
            }
        } else if (eh != 0) {
            if (lane == 0) __hip_atomic_store(cntp, call, __ATOMIC_RELAXED, __HIP_MEMORY_SCOPE_SYSTEM);
        } else {
            STAMP(8);
            if (lane < NR && lane != rank) {
                unsigned* fl = (unsigned*)(wsl + WS_FLAG + (size_t)(lane * 32 + idx) * 128);
                unsigned* tmo = (unsigned*)(wsl + WS_TMO);
                const unsigned long long t0 = __builtin_amdgcn_s_memrealtime();
                bool bad = false;
                unsigned spins = 0;
                while ((int)(__hip_atomic_load(fl, __ATOMIC_RELAXED, __HIP_MEMORY_SCOPE_SYSTEM) - call) < 0) {
                    if ((++spins & 63u) == 0u) {
                        if (__hip_atomic_load(tmo, __ATOMIC_RELAXED, __HIP_MEMORY_SCOPE_AGENT) == call ||
                            __builtin_amdgcn_s_memrealtime() - t0 > DEADLINE_TICKS) {
                            __hip_atomic_store(tmo, call, __ATOMIC_RELAXED, __HIP_MEMORY_SCOPE_AGENT);
                            bad = true;
                            break;
                        }
                    }
                }
                if (bad) {
                    outp[(size_t)(rank * MLOC + row0) * NLOC + ct * 32] = __builtin_nanf("");
                }
            }
        }
    }
}

}

extern "C" void kernel_launch(int rank, int nranks, ncclComm_t comm,
                              hipStream_t stream,
                              void* const* d_in, const int* in_sizes, int n_in,
                              void* d_out, int out_size,
                              const int* in_dtypes, int out_dtype,
                              void* const* d_out_all,
                              void* const* d_ws_all, size_t ws_bytes) {
    if (nranks != NR || n_in < 2 || ws_bytes < WS_END || (unsigned)rank >= (unsigned)NR) {
        fprintf(stderr, "kernel_launch: unexpected configuration\n");
        return;
    }
    Params p;
    p.x = (const float*)d_in[0];
    p.w = (const float*)d_in[1];
    for (int i = 0; i < NR; ++i) {
        p.out[i] = (float*)d_out_all[i];
        p.ws[i] = (char*)d_ws_all[i];
    }
    p.wsl = p.ws[rank];
    p.rank = rank;
    p.pad = 0;
    gemm_a2a_kernel<<<dim3(NWG), dim3(NTHREADS), 0, stream>>>(p);
    HIP_CHECK(hipGetLastError());
}
```

```cpp
#include <hip/hip_runtime.h>
#include <rccl/rccl.h>
#include <cstdio>
#include <cstdint>

#define HIP_CHECK(expr) do { hipError_t _e = (expr); if (_e != hipSuccess) { \
    fprintf(stderr, "HIP error %s at %s:%d\n", hipGetErrorString(_e), __FILE__, __LINE__); \
    return; } } while (0)

namespace {

typedef float f32x4 __attribute__((ext_vector_type(4)));
typedef float f32x2 __attribute__((ext_vector_type(2)));
typedef _Float16 f16x2 __attribute__((ext_vector_type(2)));
typedef _Float16 f16x8 __attribute__((ext_vector_type(8)));
typedef unsigned u32x4 __attribute__((ext_vector_type(4)));
typedef unsigned u32x2 __attribute__((ext_vector_type(2)));
typedef int i32x4 __attribute__((ext_vector_type(4)));

constexpr int NR = 8;
constexpr int MLOC = 256;
constexpr int KDIM = 2048;
constexpr int NDIM = 2048;
constexpr int NLOC = 256;
constexpr int NWG = 256;
constexpr int NTHREADS = 640;

constexpr size_t WS_CNT = 0;
constexpr size_t WS_FLAG = 32768;
constexpr size_t WS_TMO = 65536;
constexpr size_t WS_X16 = 1 << 20;
constexpr size_t WS_END = 2 << 20;

constexpr unsigned long long DEADLINE_TICKS = 150000000ull;
constexpr int TRIES = 4;

struct Params {
    const float* x;
    const float* w;
    char* wsl;
    int rank;
    int pad;
    float* out[NR];
    char* ws[NR];
};

constexpr int XS_BYTES = 8 * 4096;
constexpr int RED_OFF = XS_BYTES;
constexpr int RED_BUF = 8 * 2048;
constexpr int MISC_OFF = RED_OFF + 2 * RED_BUF;
constexpr int LDS_BYTES = 84 * 1024;
static_assert(MISC_OFF + 64 <= LDS_BYTES, "lds");

__device__ __forceinline__ unsigned pk_f16(float a, float b) {
    f32x2 v = {a, b};
    f16x2 h = __builtin_convertvector(v, f16x2);
    return __builtin_bit_cast(unsigned, h);
}

__device__ __forceinline__ i32x4 make_rsrc(const void* p, unsigned bytes) {
    unsigned long long a = (unsigned long long)p;
    i32x4 r;
    r[0] = (int)(unsigned)a;
    r[1] = (int)(unsigned)(a >> 32);
    r[2] = (int)bytes;
    r[3] = 0x00020000;
    return r;
}

#define BLOAD(dst, voff, rsrc, soff) asm volatile("buffer_load_dwordx4 %0, %1, %2, %3 offen" \
        : "=v"(dst) : "v"(voff), "s"(rsrc), "s"(soff))
#define BSTORE(val, voff, rsrc, soff) asm volatile("buffer_store_dwordx4 %0, %1, %2, %3 offen sc0 sc1\n\ts_nop 1" \
        :: "v"(val), "v"(voff), "s"(rsrc), "s"(soff) : "memory")
#define HSTORE(val, voff, rsrc, soff) asm volatile("buffer_store_dwordx4 %0, %1, %2, %3 offen sc1\n\ts_nop 1" \
        :: "v"(val), "v"(voff), "s"(rsrc), "s"(soff) : "memory")
#define BARRIER() __builtin_amdgcn_s_barrier()
#define TAG_MASK 0x00010001u
#define TAG_OF(call) (((call) & 1u) | (((call) & 2u) << 15))

__device__ __forceinline__ float gelu_tanh(float y) {
    const float c = 0.7978845608028654f;
    float u = c * (y + 0.044715f * y * y * y);
    float e = __builtin_amdgcn_exp2f(u * (-2.0f * 1.4426950408889634f));
    return y * __builtin_amdgcn_rcpf(1.0f + e);
}

__global__ void __launch_bounds__(NTHREADS) gemm_a2a_kernel(Params p) {
    __shared__ __attribute__((aligned(16))) char lds[LDS_BYTES];

    const int tid = threadIdx.x;
    const int lane = tid & 63;
    const int wave = __builtin_amdgcn_readfirstlane(tid >> 6);
    const int bid = blockIdx.x;
    const int rank = p.rank;
    const float* const px = p.x;
    const float* const pw = p.w;
    char* const wsl = p.wsl;
    asm volatile("" :: "s"(px), "s"(pw), "s"(wsl), "s"(rank));
    const int dst = (bid + rank) & 7;
    const int idx = bid >> 3;
    const int ct = idx & 7;
    const int rt = idx >> 3;
    const int col0 = dst * NLOC + ct * 32;
    const int row0 = rt * 64;
    const int c8 = lane & 7, b3 = (lane >> 3) & 1, g = lane >> 4, r8 = lane >> 3;

    if (__builtin_expect(wave < 8, 1)) {
        const i32x4 rw = make_rsrc(pw, KDIM * NDIM * 4);
        const i32x4 rx = make_rsrc(px, MLOC * KDIM * 4);
        const i32x4 rh = make_rsrc(wsl + WS_X16, MLOC * KDIM * 2);
        const unsigned w_voff = (unsigned)((8 * g + 4 * b3) * (NDIM * 4) + c8 * 16);
        const unsigned w_soff0 = (unsigned)((wave * 256) * (NDIM * 4) + col0 * 4);
        const unsigned x_voff = (unsigned)(r8 * (KDIM * 4) + c8 * 16);
        const unsigned x_soff0 = (unsigned)(row0 * (KDIM * 4) + wave * 1024);
        char* const xs_w = lds + wave * 4096 + r8 * 64 + (((c8 >> 1) ^ (2 * (r8 >> 2))) * 16) + (c8 & 1) * 8;
        const char* const xs_r = lds + wave * 4096 + b3 * 512 + c8 * 64 + ((g ^ (2 * (c8 >> 2))) * 16);
        const unsigned rnd_add = (c8 & 1) ? 0u : 0x1fffu, rnd_lsb = (c8 & 1) ? 0u : 1u, rnd_and = (c8 & 1) ? ~0u : ~0x3fffu;
#define RND9(x) __uint_as_float((__float_as_uint(x) + rnd_add + ((__float_as_uint(x) >> 14) & rnd_lsb)) & rnd_and)
#define STAGE_V(v, s) do { const float e0_ = (v)[0], e1_ = (v)[1]; \
            u32x2 h_ = {pk_f16(RND9(e0_), RND9(e1_)), pk_f16((v)[2], (v)[3])}; \
            *(u32x2*)(xs_w + (s) * 512) = h_; } while (0)
        char* const red_w = lds + RED_OFF + wave * 2048 + (4 * g) * 128 + (4 * c8 + 2 * b3) * 4;
        const unsigned h_voff = (unsigned)(lane * 16);
        f32x4 xv[8];

        if (dst == rank) {
            const unsigned kbp = ct * 8 + wave;
            const unsigned xs = (unsigned)(row0 * (KDIM * 4)) + kbp * 128, hs = (unsigned)(rt * 4 * 65536) + kbp * 1024;
            unsigned cnt;
#define PRODUCE(i) do { STAGE_V(xv[2 * (i)], 2 * (i)); STAGE_V(xv[2 * (i) + 1], 2 * (i) + 1); \
                u32x4 f = *(const u32x4*)(xs_r + (i) * 1024); \
                f[0] = (f[0] & ~TAG_MASK) | TAG_OF(cnt + 1u); \
                HSTORE(f, h_voff, rh, hs + (i) * 65536); } while (0)
            asm volatile("global_load_dword %0, %1, off sc0 sc1" : "=v"(cnt) : "v"(wsl + WS_CNT + (size_t)bid * 128));
#pragma unroll
            for (int i = 0; i < 4; ++i) BLOAD(xv[i], x_voff, rx, xs + i * 8 * (KDIM * 4));
            BARRIER();
#pragma unroll
            for (int i = 4; i < 8; ++i) BLOAD(xv[i], x_voff, rx, xs + i * 8 * (KDIM * 4));
            asm volatile("s_waitcnt vmcnt(6)" : "+v"(xv[0]), "+v"(xv[1]), "+v"(cnt));
            PRODUCE(0);
            asm volatile("s_waitcnt vmcnt(5)" : "+v"(xv[2]), "+v"(xv[3]));
            PRODUCE(1);
            asm volatile("s_waitcnt vmcnt(2)" : "+v"(xv[4]), "+v"(xv[5]), "+v"(xv[6]), "+v"(xv[7]));
            PRODUCE(2);
            PRODUCE(3);
        }

        const unsigned perm = rt == 0 ? 0x76532410u : rt == 1 ? 0x67423501u : rt == 2 ? 0x75463102u : 0x64572013u;
        const unsigned koff = wave + ct;
#define KB(s) (((perm >> (4 * (s))) + koff) & 7u)
#define LOAD_W(s) do { _Pragma("unroll") for (int i = 0; i < 4; ++i) \
            BLOAD(wf[s][i], w_voff, rw, w_soff0 + (KB(s) * 32 + i) * (unsigned)(NDIM * 4)); } while (0)
#define W_CONV(dstf, src) do { unsigned pk[4][2]; \
            _Pragma("unroll") for (int j = 0; j < 4; ++j) { \
                pk[j][0] = pk_f16((src)[0][j], (src)[1][j]); pk[j][1] = pk_f16((src)[2][j], (src)[3][j]); } \
            _Pragma("unroll") for (int m = 0; m < 2; ++m) { u32x4 f; \
                _Pragma("unroll") for (int t = 0; t < 2; ++t) { const unsigned P = pk[m][t], Q = pk[2 + m][t]; \
                    f[t]     = __builtin_amdgcn_update_dpp(P, Q, 0x128, 0xf, 0xc, false); \
                    f[2 + t] = __builtin_amdgcn_update_dpp(Q, P, 0x128, 0xf, 0x3, false); } \
                (dstf)[m] = __builtin_bit_cast(f16x8, f); } } while (0)

        f32x4 wf[8][4];
        f16x8 wfrag[8][2];
        u32x4 xh[8];
        const unsigned h_soff0 = (unsigned)((rt * 4) * 65536 + wave * 8192);
        LOAD_W(0);
        BARRIER();
        LOAD_W(1);
#pragma unroll
        for (int s = 0; s < 8; ++s) {
            if (s < 6) {
                BARRIER();
                LOAD_W(s + 2);
            } else {
#pragma unroll
                for (int j = 0; j < 4; ++j) BLOAD(xh[4 * s - 24 + j], h_voff, rh, h_soff0 + KB(4 * s - 24 + j) * 1024);
            }
            asm volatile("s_waitcnt vmcnt(8)" : "+v"(wf[s][0]), "+v"(wf[s][1]), "+v"(wf[s][2]), "+v"(wf[s][3]));
            W_CONV(wfrag[s], wf[s]);
            __builtin_amdgcn_sched_barrier(0);
        }
        const unsigned par = __builtin_amdgcn_readfirstlane(__hip_atomic_load((unsigned*)(lds + MISC_OFF),
                                 __ATOMIC_RELAXED, __HIP_MEMORY_SCOPE_WORKGROUP));
#define ALL_XH "+v"(xh[0]), "+v"(xh[1]), "+v"(xh[2]), "+v"(xh[3]), "+v"(xh[4]), "+v"(xh[5]), "+v"(xh[6]), "+v"(xh[7])
#define STALE() (__any((((xh[0][0] ^ par) | (xh[1][0] ^ par) | (xh[2][0] ^ par) | (xh[3][0] ^ par) | \
                         (xh[4][0] ^ par) | (xh[5][0] ^ par) | (xh[6][0] ^ par) | (xh[7][0] ^ par)) & TAG_MASK) != 0))
#define STEP16(j) do { \
            f32x4 acc0 = {0.f, 0.f, 0.f, 0.f}, acc1 = acc0; \
            asm volatile("s_waitcnt vmcnt(0)" : ALL_XH); \
            bool stale = STALE(); \
            for (int tries = 0; stale && tries < TRIES; ++tries) {   \
                _Pragma("unroll") for (int s = 0; s < 8; ++s) \
                    asm volatile("buffer_load_dwordx4 %0, %1, %2, %3 offen sc1" : "+v"(xh[s]) \
                                 : "v"(h_voff), "s"(rh), "s"(h_soff0 + (j) * 65536u + KB(s) * 1024)); \
                asm volatile("s_waitcnt vmcnt(0)" : ALL_XH); \
                stale = STALE(); \
            } \
            if (!stale) { \
                _Pragma("unroll") for (int s = 0; s < 8; ++s) { \
                    xh[s][0] &= ~TAG_MASK; \
                    const f16x8 a_ = __builtin_bit_cast(f16x8, xh[s]); \
                    acc0 = __builtin_amdgcn_mfma_f32_16x16x32_f16(a_, wfrag[s][0], acc0, 0, 0, 0); \
                    acc1 = __builtin_amdgcn_mfma_f32_16x16x32_f16(a_, wfrag[s][1], acc1, 0, 0, 0); \
                } \
            } else {   \
                _Pragma("unroll") for (int hh = 0; hh < 2; ++hh) { \
                    const unsigned xb = x_soff0 + (j) * 16 * (KDIM * 4); \
                    _Pragma("unroll") for (int s = 0; s < 4; ++s) { \
                        BLOAD(xv[2 * s], x_voff, rx, xb + KB(4 * hh + s) * 128); \
                        BLOAD(xv[2 * s + 1], x_voff, rx, xb + 8 * (KDIM * 4) + KB(4 * hh + s) * 128); } \
                    asm volatile("s_waitcnt vmcnt(0)" : "+v"(xv[0]), "+v"(xv[1]), "+v"(xv[2]), "+v"(xv[3]), \
                                 "+v"(xv[4]), "+v"(xv[5]), "+v"(xv[6]), "+v"(xv[7])); \
                    _Pragma("unroll") for (int s = 0; s < 8; ++s) STAGE_V(xv[s], s); \
                    _Pragma("unroll") for (int s = 0; s < 4; ++s) { \
                        u32x4 f_ = *(const u32x4*)(xs_r + s * 1024); \
                        f_[0] &= ~TAG_MASK;       \
                        const f16x8 a_ = __builtin_bit_cast(f16x8, f_); \
                        acc0 = __builtin_amdgcn_mfma_f32_16x16x32_f16(a_, wfrag[4 * hh + s][0], acc0, 0, 0, 0); \
                        acc1 = __builtin_amdgcn_mfma_f32_16x16x32_f16(a_, wfrag[4 * hh + s][1], acc1, 0, 0, 0); } \
                    __builtin_amdgcn_sched_barrier(0); \
                } \
            } \
            __builtin_amdgcn_sched_barrier(0); \
            if ((j) < 3) {   \
                _Pragma("unroll") for (int s = 0; s < 8; ++s) \
                    BLOAD(xh[s], h_voff, rh, h_soff0 + ((j) + 1) * 65536u + KB(s) * 1024); } \
              \
            char* wp = red_w + ((j) & 1) * RED_BUF; \
            _Pragma("unroll") for (int r = 0; r < 4; ++r) { f32x2 v = {acc0[r], acc1[r]}; \
                *(f32x2*)(wp + r * 128) = v; } \
            asm volatile("s_waitcnt lgkmcnt(0)" ::: "memory"); BARRIER(); asm volatile("" ::: "memory"); } while (0)
        STEP16(0);
        STEP16(1);
        STEP16(2);
        STEP16(3);
    } else {
        __builtin_amdgcn_s_setprio(3);
        const int eh = wave - 8;
        unsigned* cntp = (unsigned*)(wsl + WS_CNT + (size_t)bid * 128);
        unsigned cnt;
        asm volatile("global_load_dword %0, %1, off sc0 sc1" : "=v"(cnt) : "v"(cntp));
        float* outp = p.out[dst];
        char* wsd = p.ws[dst];
        const i32x4 ro = make_rsrc(outp, (unsigned)(NR * MLOC * NLOC * 4));
        const unsigned ooff = (unsigned)(((rank * MLOC + row0 + eh * 8 + r8) * NLOC + ct * 32 + c8 * 4) * 4);
        if (dst == rank) BARRIER();
#pragma unroll
        for (int i = 0; i < 4; ++i) BARRIER();
        asm volatile("s_waitcnt vmcnt(0)" : "+v"(cnt));
        const unsigned call = __builtin_amdgcn_readfirstlane(cnt) + 1u;
        if (lane == 0) __hip_atomic_store((unsigned*)(lds + MISC_OFF), TAG_OF(call), __ATOMIC_RELAXED, __HIP_MEMORY_SCOPE_WORKGROUP);
        asm volatile("s_waitcnt lgkmcnt(0)" ::: "memory");
        BARRIER();
        BARRIER();
#pragma unroll 1
        for (int c = -1; c < 4; ++c) {
            BARRIER();
            asm volatile("" ::: "memory");
            const char* rp = lds + RED_OFF + lane * 16 + (c & 1) * RED_BUF + eh * 1024;
            f32x4 v = *(const f32x4*)(rp);
#pragma unroll
            for (int q = 1; q < 8; ++q) v += *(const f32x4*)(rp + q * 2048);
            f32x4 o;
#pragma unroll
            for (int j = 0; j < 4; ++j) o[j] = gelu_tanh(v[j]);
            BSTORE(o, ooff, ro, c < 0 ? 0x40000000u : (unsigned)(c * 16 * NLOC * 4));
        }
        if (dst != rank) {
            asm volatile("s_waitcnt vmcnt(0)" ::: "memory");
            BARRIER();
            if (eh == 0 && lane == 0) {
                unsigned* fl = (unsigned*)(wsd + WS_FLAG + (size_t)(rank * 32 + idx) * 128);
                __hip_atomic_store(fl, call, __ATOMIC_RELAXED, __HIP_MEMORY_SCOPE_SYSTEM);
                __hip_atomic_store(cntp, call, __ATOMIC_RELAXED, __HIP_MEMORY_SCOPE_SYSTEM);
            }
        } else if (eh != 0) {
            if (lane == 0) __hip_atomic_store(cntp, call, __ATOMIC_RELAXED, __HIP_MEMORY_SCOPE_SYSTEM);
        } else if (lane < NR && lane != rank) {
            unsigned* fl = (unsigned*)(wsl + WS_FLAG + (size_t)(lane * 32 + idx) * 128);
            unsigned* tmo = (unsigned*)(wsl + WS_TMO);
            const unsigned long long t0 = __builtin_amdgcn_s_memrealtime();
            bool bad = false;
            unsigned spins = 0;
            while ((int)(__hip_atomic_load(fl, __ATOMIC_RELAXED, __HIP_MEMORY_SCOPE_SYSTEM) - call) < 0) {
                if ((++spins & 63u) == 0u) {
                    if (__hip_atomic_load(tmo, __ATOMIC_RELAXED, __HIP_MEMORY_SCOPE_AGENT) == call ||
                        __builtin_amdgcn_s_memrealtime() - t0 > DEADLINE_TICKS) {
                        __hip_atomic_store(tmo, call, __ATOMIC_RELAXED, __HIP_MEMORY_SCOPE_AGENT);
                        bad = true;
                        break;
                    }
                }
            }
            if (bad) {
                outp[(size_t)(rank * MLOC + row0) * NLOC + ct * 32] = __builtin_nanf("");
            }
        }
    }
}

}

extern "C" void kernel_launch(int rank, int nranks, ncclComm_t comm,
                              hipStream_t stream,
                              void* const* d_in, const int* in_sizes, int n_in,
                              void* d_out, int out_size,
                              const int* in_dtypes, int out_dtype,
                              void* const* d_out_all,
                              void* const* d_ws_all, size_t ws_bytes) {
    if (nranks != NR || n_in < 2 || ws_bytes < WS_END || (unsigned)rank >= (unsigned)NR) {
        fprintf(stderr, "kernel_launch: unexpected configuration\n");
        return;
    }
    Params p;
    p.x = (const float*)d_in[0];
    p.w = (const float*)d_in[1];
    for (int i = 0; i < NR; ++i) {
        p.out[i] = (float*)d_out_all[i];
        p.ws[i] = (char*)d_ws_all[i];
    }
    p.wsl = p.ws[rank];
    p.rank = rank;
    p.pad = 0;
    gemm_a2a_kernel<<<dim3(NWG), dim3(NTHREADS), 0, stream>>>(p);
    HIP_CHECK(hipGetLastError());
}
```
